```python
import numpy as np

kernel_name = "gpu_gemm_ar_m2048_k2048_n2048_bf16_silu"


ABSMAX_RTOL = 0.008

COMPUTE_DTYPE = "bfloat16"
EPILOGUE = "silu"
IN_QUANT = None
OUT_QUANT = None
IN_DTYPES = {"x": "bfloat16", "w_mat": "bfloat16"}
WORLD_SIZE = 8

def _to_bf16(x):
    u = np.ascontiguousarray(x, dtype=np.float32).view(np.uint32)
    return ((u + 0x7FFF + ((u >> 16) & 1)) & 0xFFFF0000).view(np.float32)

def setup_inputs():

    x = _to_bf16(np.random.randn(2048, 2048).astype(np.float32))
    w_mat = _to_bf16((np.random.randn(2048, 2048) / np.sqrt(2048)).astype(np.float32))
    return {"x": x, "w_mat": w_mat}

def _epilogue(y):
    return (y / (1.0 + np.exp(-np.clip(y, -60.0, 60.0)))).astype(np.float32)

def reference(x, w_mat):
    y = (x @ w_mat).astype(np.float32)
    return _epilogue(y)
```

```cpp
#include <hip/hip_runtime.h>
#include <rccl/rccl.h>
#include <cstdio>
#include <cstdint>

#define HIP_CHECK(expr) do { hipError_t _e = (expr); if (_e != hipSuccess) { \
    fprintf(stderr, "HIP error %s at %s:%d\n", hipGetErrorString(_e), __FILE__, __LINE__); \
    return; } } while (0)

typedef unsigned int u32;
typedef unsigned long long u64;
typedef unsigned short u16;
typedef __attribute__((ext_vector_type(4))) unsigned int u32x4;
typedef __attribute__((ext_vector_type(2))) unsigned int u32x2;
typedef __attribute__((ext_vector_type(8))) short bf16x8;
typedef __attribute__((ext_vector_type(4))) float f32x4;

#define NPUSH 32
constexpr int NR = 8;
constexpr int MM = 2048, NN = 2048, KSH = 256;
constexpr int NTASK = 96;
constexpr int NT = NTASK / NPUSH;
constexpr int NGEMM = 128;
constexpr int GRID = NPUSH + NGEMM;
static_assert(NT * NPUSH == NTASK, "NPUSH must divide 96");
static_assert(8 * (NT - 1) + NT <= 63, "vmcnt range");

constexpr size_t XS_PER = 512u << 10;
constexpr size_t WS_PER = 256u << 10;
constexpr size_t XS_OFF = 0;
constexpr size_t WS_OFF = NR * XS_PER;
constexpr size_t SLOT_BYTES = NR * (XS_PER + WS_PER);
constexpr size_t FLAG_OFF = 2 * SLOT_BYTES;
constexpr size_t FSTR = 128;
constexpr size_t WF_OFF = FLAG_OFF;
constexpr size_t XF_OFF = WF_OFF + 64 * FSTR;
constexpr size_t OF_OFF = XF_OFF + 256 * FSTR;
constexpr size_t CNT_OFF = OF_OFF + (size_t)NR * NGEMM * FSTR;
constexpr size_t TMO_OFF = CNT_OFF + 256 * FSTR;
constexpr size_t WS_NEEDED = TMO_OFF + FSTR;

constexpr u64 DEADLINE_TICKS = 40000000ull;

struct Params {
  const u16* x;
  const u16* w;
  char* out[NR];
  char* ws[NR];
  int rank;
};

constexpr int AUX_SYS = 17;

__device__ __forceinline__ __amdgpu_buffer_rsrc_t mk_rsrc(const void* p, u32 bytes) {
  return __builtin_amdgcn_make_buffer_rsrc((void*)p, 0, bytes, 0x00020000);
}
__device__ __forceinline__ void st_sys(u32x4 v, __amdgpu_buffer_rsrc_t r, u32 off) {
  __builtin_amdgcn_raw_buffer_store_b128(v, r, off, 0, AUX_SYS);
}
__device__ __forceinline__ u32x4 ld_sys(__amdgpu_buffer_rsrc_t r, u32 off) {
  return __builtin_amdgcn_raw_buffer_load_b128(r, off, 0, AUX_SYS);
}

__device__ __forceinline__ bool wait_flag(const u32* f, u32 call, u32* tmo) {
  u64 t0 = 0;
  u32 spins = 0;
  for (;;) {
    const u32 v = __hip_atomic_load(f, __ATOMIC_RELAXED, __HIP_MEMORY_SCOPE_SYSTEM);
    if (__all(v >= call)) break;
    if ((++spins & 31u) == 0) {
      const u64 now = __builtin_amdgcn_s_memrealtime();
      if (t0 == 0) t0 = now;
      if (__hip_atomic_load(tmo, __ATOMIC_RELAXED, __HIP_MEMORY_SCOPE_AGENT) == call ||
          now - t0 > DEADLINE_TICKS) {
        __hip_atomic_store(tmo, call, __ATOMIC_RELAXED, __HIP_MEMORY_SCOPE_AGENT);
        return false;
      }
    }
    __builtin_amdgcn_s_sleep(1);
  }
  asm volatile("" ::: "memory");
  return true;
}

template <int N> __device__ __forceinline__ void wait_vm() {
  asm volatile("s_waitcnt vmcnt(%0)" :: "i"(N) : "memory");
}

__device__ __forceinline__ u32 pack_bf16(float a, float b) {
  u32 r;
  asm("v_cvt_pk_bf16_f32 %0, %1, %2" : "=v"(r) : "v"(a), "v"(b));
  return r;
}
__device__ __forceinline__ float silu(float y) {
  const float yc = fminf(fmaxf(y, -60.0f), 60.0f);
  return y / (1.0f + __expf(-yc));
}

constexpr int PROW = 272;
constexpr int PART_BYTES = 32 * PROW;
constexpr int LDS_BYTES = 8 * PART_BYTES + 64;

template <int n>
__device__ __forceinline__ void push_signal(u32* const (&fp)[NT], const bool (&fa)[NT], int wv, u32 call) {
  if (wv == 0) wait_vm<8 * (NT - 1 - n) + n>(); else wait_vm<8 * (NT - 1 - n)>();
  __builtin_amdgcn_s_barrier();
  if (wv == 0) {
    if (fa[n]) __hip_atomic_store(fp[n], call, __ATOMIC_RELAXED, __HIP_MEMORY_SCOPE_SYSTEM);
  }
  if constexpr (n + 1 < NT) push_signal<n + 1>(fp, fa, wv, call);
}

__device__ __forceinline__ char* pick4(char* a, char* b, char* c, char* d, int sel) {
  char* lo = (sel & 1) ? b : a;
  char* hi = (sel & 1) ? d : c;
  return (sel & 2) ? hi : lo;
}

__global__ void __launch_bounds__(512, 2) gemm_ar_silu(Params p) {
  __shared__ __attribute__((aligned(16))) char smem[LDS_BYTES];
  const int tid = threadIdx.x, lane = tid & 63;
  const int wv = __builtin_amdgcn_readfirstlane(tid >> 6);
  const int bid = blockIdx.x;
  const int rank = p.rank;
  char* myws = p.ws[rank];
  u32* cntp = (u32*)(myws + CNT_OFF + (size_t)bid * FSTR);
  u32* tmo = (u32*)(myws + TMO_OFF);

  if (bid < NPUSH) {
    const int pid = bid;
    const u32 cnt = __hip_atomic_load(cntp, __ATOMIC_RELAXED, __HIP_MEMORY_SCOPE_AGENT);
    const int c = lane & 15, kg = lane >> 4;
    u32 r[NT][16];
#pragma unroll
    for (int n = 0; n < NT; ++n) {
      const int t = pid + n * NPUSH;
      if (t < 32) {
        const u16* src = p.w + (size_t)(32 * wv + 8 * kg) * NN + 64 * t + 4 * c;
#pragma unroll
        for (int jj = 0; jj < 8; ++jj) {
          const u32x2 v = *(const u32x2*)(src + (size_t)jj * NN);
          r[n][2 * jj] = v.x; r[n][2 * jj + 1] = v.y;
        }
      } else {
        const int u = t - 32, b = u >> 1, i = u & 1;
        const u16* src = p.x + (size_t)(1024 * i + 32 * b + c) * KSH + 32 * wv + 8 * kg;
#pragma unroll
        for (int rf = 0; rf < 2; ++rf) {
          const u32x4 v = *(const u32x4*)(src + (size_t)rf * 16 * KSH);
          r[n][4 * rf] = v.x; r[n][4 * rf + 1] = v.y; r[n][4 * rf + 2] = v.z; r[n][4 * rf + 3] = v.w;
        }
      }
    }
    u32* fp[NT]; bool fa[NT];
#pragma unroll
    for (int n = 0; n < NT; ++n) {
      const int t = pid + n * NPUSH;
      if (t < 32) {
        const int j = t >> 3, g = t & 7;
        char* base = (lane & 1) ? p.ws[4 + j] : p.ws[j];
        fp[n] = (u32*)(base + WF_OFF + (size_t)(rank * 8 + g) * FSTR); fa[n] = lane < 2;
      } else {
        const int u = t - 32, b = u >> 1, i = u & 1;
        char* base = pick4(p.ws[4 * i], p.ws[4 * i + 1], p.ws[4 * i + 2], p.ws[4 * i + 3], lane & 3);
        fp[n] = (u32*)(base + XF_OFF + (size_t)(rank * 32 + b) * FSTR); fa[n] = lane < 4;
      }
    }
    const u32 call = __builtin_amdgcn_readfirstlane(cnt) + 1u;
    const size_t slot = (call & 1u) ? SLOT_BYTES : 0;
#pragma unroll
    for (int n = 0; n < NT; ++n) {
      const int t = pid + n * NPUSH;
      if (t < 32) {
        const int j = t >> 3, g = t & 7;
        u32x4 f[4];
#pragma unroll
        for (int d = 0; d < 4; ++d) {
          const u32 a0 = r[n][4 * d], a1 = r[n][4 * d + 1], b0 = r[n][4 * d + 2], b1 = r[n][4 * d + 3];
          f[0][d] = __builtin_amdgcn_perm(b0, a0, 0x05040100u);
          f[1][d] = __builtin_amdgcn_perm(b0, a0, 0x07060302u);
          f[2][d] = __builtin_amdgcn_perm(b1, a1, 0x05040100u);
          f[3][d] = __builtin_amdgcn_perm(b1, a1, 0x07060302u);
        }
        const u32 off = (u32)(((g * 8 + wv) * 4) * 1024 + lane * 16);
#pragma unroll
        for (int ii = 0; ii < 2; ++ii) {
          const int dst = 4 * ((ii + wv) & 1) + j;
          const __amdgpu_buffer_rsrc_t rs = mk_rsrc(p.ws[dst] + slot + WS_OFF + (size_t)rank * WS_PER, (u32)WS_PER);
#pragma unroll
          for (int ff = 0; ff < 4; ++ff) st_sys(f[ff], rs, off + ff * 1024);
        }
      } else {
        const int u = t - 32, b = u >> 1, i = u & 1;
        const u32 off = (u32)(((b * 2) * 8 + wv) * 1024 + lane * 16);
#pragma unroll
        for (int jj = 0; jj < 4; ++jj) {
          const int dst = 4 * i + ((jj + wv) & 3);
          const __amdgpu_buffer_rsrc_t rs = mk_rsrc(p.ws[dst] + slot + XS_OFF + (size_t)rank * XS_PER, (u32)XS_PER);
#pragma unroll
          for (int rf = 0; rf < 2; ++rf) {
            const u32x4 v = {r[n][4 * rf], r[n][4 * rf + 1], r[n][4 * rf + 2], r[n][4 * rf + 3]};
            st_sys(v, rs, off + rf * 8192);
          }
        }
      }
    }
    asm volatile("" ::: "memory");
    push_signal<0>(fp, fa, wv, call);
    if (tid == 0) __hip_atomic_store(cntp, call, __ATOMIC_RELAXED, __HIP_MEMORY_SCOPE_AGENT);
    return;
  }

  const int gid = bid - NPUSH;
  const int g = gid & 7, bp = gid >> 3;
  const int ri = rank >> 2, rj = rank & 3;
  const u32 cnt = __hip_atomic_load(cntp, __ATOMIC_RELAXED, __HIP_MEMORY_SCOPE_AGENT);
  const u32 call = __builtin_amdgcn_readfirstlane(cnt) + 1u;
  const size_t slot = (call & 1u) ? SLOT_BYTES : 0;
  const int q = wv;

  bool ok = wait_flag((const u32*)(myws + WF_OFF + (size_t)(q * 8 + g) * FSTR), call, tmo);
  const __amdgpu_buffer_rsrc_t wr = mk_rsrc(myws + slot + WS_OFF + (size_t)q * WS_PER, (u32)WS_PER);
  const __amdgpu_buffer_rsrc_t xr = mk_rsrc(myws + slot + XS_OFF + (size_t)q * XS_PER, (u32)XS_PER);
  u32x4 wf[8][4];
#pragma unroll
  for (int s = 0; s < 8; ++s)
#pragma unroll
    for (int f = 0; f < 4; ++f)
      wf[s][f] = ld_sys(wr, (u32)(((g * 8 + s) * 4 + f) * 1024 + lane * 16));

  u32* sbad = (u32*)(smem + 8 * PART_BYTES);
  const int half = wv >> 2;
  const int t256 = tid & 255;
  const int orow = t256 >> 3, oseg = t256 & 7;

  u32x4 xf[2][8];
  f32x4 acc[2][4];
  auto load_x = [&](int b) {
#pragma unroll
    for (int s = 0; s < 8; ++s)
#pragma unroll
      for (int rf = 0; rf < 2; ++rf)
        xf[rf][s] = ld_sys(xr, (u32)(((b * 2 + rf) * 8 + s) * 1024 + lane * 16));
    __builtin_amdgcn_sched_barrier(0);
  };
  auto mma = [&]() {
#pragma unroll
    for (int rf = 0; rf < 2; ++rf)
#pragma unroll
      for (int f = 0; f < 4; ++f) acc[rf][f] = f32x4{0.f, 0.f, 0.f, 0.f};
#pragma unroll
    for (int s = 0; s < 8; ++s)
#pragma unroll
      for (int rf = 0; rf < 2; ++rf)
#pragma unroll
        for (int f = 0; f < 4; ++f)
          acc[rf][f] = __builtin_amdgcn_mfma_f32_16x16x32_bf16(
              __builtin_bit_cast(bf16x8, wf[s][f]), __builtin_bit_cast(bf16x8, xf[rf][s]), acc[rf][f], 0, 0, 0);
    __builtin_amdgcn_sched_barrier(0);
  };
  auto finish = [&](int b) {
    {
      char* part = smem + wv * PART_BYTES;
      const int m = lane & 15, q4 = lane >> 4;
#pragma unroll
      for (int rf = 0; rf < 2; ++rf)
#pragma unroll
        for (int reg = 0; reg < 4; ++reg) {
          const f32x4 v = {acc[rf][0][reg], acc[rf][1][reg], acc[rf][2][reg], acc[rf][3][reg]};
          *(f32x4*)(part + (16 * rf + m) * PROW + q4 * 64 + reg * 16) = v;
        }
      if (lane == 0) sbad[wv] = ok ? 0u : 1u;
    }
    asm volatile("s_waitcnt lgkmcnt(0)" ::: "memory");
    __builtin_amdgcn_s_barrier();
    asm volatile("" ::: "memory");
    f32x4 s0 = {0.f, 0.f, 0.f, 0.f}, s1 = {0.f, 0.f, 0.f, 0.f};
#pragma unroll
    for (int k = 0; k < 8; ++k) {
      const char* pp = smem + k * PART_BYTES + orow * PROW + oseg * 32;
      s0 += *(const f32x4*)pp;
      s1 += *(const f32x4*)(pp + 16);
    }
    const u32x4 bad0 = *(const u32x4*)sbad, bad1 = *(const u32x4*)(sbad + 4);
    const bool bad = (bad0.x | bad0.y | bad0.z | bad0.w | bad1.x | bad1.y | bad1.z | bad1.w) != 0u;
    u32x4 o;
    o.x = pack_bf16(silu(s0[0]), silu(s0[1]));
    o.y = pack_bf16(silu(s0[2]), silu(s0[3]));
    o.z = pack_bf16(silu(s1[0]), silu(s1[1]));
    o.w = pack_bf16(silu(s1[2]), silu(s1[3]));
    if (bad) o = u32x4{0x7fc07fc0u, 0x7fc07fc0u, 0x7fc07fc0u, 0x7fc07fc0u};
    const u32 ooff = (u32)(((1024 * ri + 32 * b + orow) * NN + 512 * rj + 64 * g + 8 * oseg) * 2);
#pragma unroll
    for (int d = 0; d < 4; ++d) {
      const int dst = 4 * half + ((d + gid) & 3);
      const __amdgpu_buffer_rsrc_t rs = mk_rsrc(p.out[dst], (u32)(MM * NN * 2));
      st_sys(o, rs, ooff);
    }
    __builtin_amdgcn_sched_barrier(0);
  };

  const int bA = 2 * bp, bB = 2 * bp + 1;
  ok = wait_flag((const u32*)(myws + XF_OFF + (size_t)(q * 32 + bA) * FSTR), call, tmo) && ok;
  load_x(bA);
  mma();
  ok = wait_flag((const u32*)(myws + XF_OFF + (size_t)(q * 32 + bB) * FSTR), call, tmo) && ok;
  load_x(bB);
  finish(bA);
  mma();
  asm volatile("s_waitcnt lgkmcnt(0)" ::: "memory");
  __builtin_amdgcn_s_barrier();
  asm volatile("" ::: "memory");
  finish(bB);

  wait_vm<0>();
  __builtin_amdgcn_s_barrier();
  if (wv == 0) {
    const int pr = lane & 7;
    char* lo = pick4(p.ws[0], p.ws[1], p.ws[2], p.ws[3], pr);
    char* hi = pick4(p.ws[4], p.ws[5], p.ws[6], p.ws[7], pr);
    char* pws = (pr & 4) ? hi : lo;
    if (lane < NR && pr != rank)
      __hip_atomic_store((u32*)(pws + OF_OFF + (size_t)(rank * NGEMM + gid) * FSTR), call,
                         __ATOMIC_RELAXED, __HIP_MEMORY_SCOPE_SYSTEM);
    bool ok2 = true;
    if (lane < NR && pr != rank)
      ok2 = wait_flag((const u32*)(myws + OF_OFF + (size_t)(pr * NGEMM + gid) * FSTR), call, tmo);
    if (!__all(ok2) && lane == 0) {
      u16* o16 = (u16*)p.out[rank];
      o16[(size_t)(1024 * ri + 32 * bA) * NN + 512 * rj + 64 * g] = 0x7fc0;
    }
    if (lane == 0) __hip_atomic_store(cntp, call, __ATOMIC_RELAXED, __HIP_MEMORY_SCOPE_AGENT);
  }
}

extern "C" void kernel_launch(int rank, int nranks, ncclComm_t comm,
                              hipStream_t stream,
                              void* const* d_in, const int* in_sizes, int n_in,
                              void* d_out, int out_size,
                              const int* in_dtypes, int out_dtype,
                              void* const* d_out_all,
                              void* const* d_ws_all, size_t ws_bytes) {
  (void)comm; (void)in_sizes; (void)n_in; (void)d_out; (void)out_size; (void)in_dtypes; (void)out_dtype;
  if (nranks != NR || (unsigned)rank >= (unsigned)NR || ws_bytes < WS_NEEDED) {
    fprintf(stderr, "unsupported configuration\n");
    return;
  }
  Params p;
  p.x = (const u16*)d_in[0];
  p.w = (const u16*)d_in[1];
  for (int i = 0; i < NR; ++i) { p.out[i] = (char*)d_out_all[i]; p.ws[i] = (char*)d_ws_all[i]; }
  p.rank = rank;
  gemm_ar_silu<<<GRID, 512, 0, stream>>>(p);
  HIP_CHECK(hipGetLastError());
}
```

```cpp
#include <hip/hip_runtime.h>
#include <rccl/rccl.h>
#include <cstdio>
#include <cstdint>

#define HIP_CHECK(expr) do { hipError_t _e = (expr); if (_e != hipSuccess) { \
    fprintf(stderr, "HIP error %s at %s:%d\n", hipGetErrorString(_e), __FILE__, __LINE__); \
    return; } } while (0)

typedef unsigned int u32;
typedef unsigned long long u64;
typedef unsigned short u16;
typedef __attribute__((ext_vector_type(4))) unsigned int u32x4;
typedef __attribute__((ext_vector_type(2))) unsigned int u32x2;
typedef __attribute__((ext_vector_type(8))) short bf16x8;
typedef __attribute__((ext_vector_type(4))) float f32x4;

#define NPUSH 64
#define ND 0
#define PSLEEP 2
#define PDB 2
#define GATE 1
#define OWAIT 1
#define LA 1
constexpr int NR = 8;
constexpr int MM = 2048, NN = 2048, KSH = 256;
constexpr int PPD = NPUSH / NR;
constexpr int NWT = 16;
constexpr int NXT = 32;
constexpr int NTASK = NWT + NXT;
constexpr int NT = NTASK / PPD;
constexpr int NGEMM = 128;
constexpr int GRID = NPUSH + NGEMM;
static_assert(NT * PPD == NTASK && PPD * NR == NPUSH, "NPUSH/8 must divide the task count");
static_assert(NT <= 8, "registers");

constexpr size_t XS_PER = 512u << 10;
constexpr size_t WS_PER = 256u << 10;
constexpr size_t XS_OFF = 0;
constexpr size_t WS_OFF = NR * XS_PER;
constexpr size_t SLOT_BYTES = NR * (XS_PER + WS_PER);
constexpr size_t FLAG_OFF = SLOT_BYTES;
constexpr size_t FSTR = 128;
constexpr size_t WF_OFF = FLAG_OFF;
constexpr size_t XF_OFF = WF_OFF + 8 * FSTR;
constexpr size_t TF_OFF = XF_OFF + 16 * FSTR;
constexpr size_t DF_OFF = TF_OFF + 32 * FSTR;
constexpr size_t OF_OFF = DF_OFF + 32 * FSTR;
constexpr size_t CNT_OFF = OF_OFF + 256 * FSTR;
constexpr size_t TMO_OFF = CNT_OFF + (size_t)GRID * FSTR;
constexpr size_t WS_NEEDED = TMO_OFF + FSTR;
static_assert(PPD == 8, "output task mapping assumes 8 engine workgroups per link");

constexpr u64 DEADLINE_TICKS = 40000000ull;

#define TS(i) do {} while (0)
#define TS_FLUSH() do {} while (0)
#define TS_ARG
#define TS_PASS

struct Params {
  const u16* x;
  const u16* w;
  char* out[NR];
  char* ws[NR];
  int rank;
};

constexpr int AUX_SYS = 17;
#define AUX_LD 17

__device__ __forceinline__ __amdgpu_buffer_rsrc_t mk_rsrc(const void* p, u32 bytes) {
  return __builtin_amdgcn_make_buffer_rsrc((void*)p, 0, bytes, 0x00020000);
}
__device__ __forceinline__ void st_sys(u32x4 v, __amdgpu_buffer_rsrc_t r, u32 off) {
  __builtin_amdgcn_raw_buffer_store_b128(v, r, off, 0, AUX_SYS);
}
__device__ __forceinline__ u32x4 ld_sys(__amdgpu_buffer_rsrc_t r, u32 off) {
  return __builtin_amdgcn_raw_buffer_load_b128(r, off, 0, AUX_LD);
}

typedef __attribute__((address_space(1))) u32 gu32;
__device__ __forceinline__ void st_flag(u32* f, u32 v) {
  __hip_atomic_store((gu32*)(size_t)f, v, __ATOMIC_RELAXED, __HIP_MEMORY_SCOPE_SYSTEM);
}

__device__ __forceinline__ bool wait_ge(const u32* f, u32 thr, u32 call, u32* tmo) {
  u64 t0 = 0;
  u32 spins = 0;
  for (;;) {
    const u32 v = __hip_atomic_load(f, __ATOMIC_RELAXED, __HIP_MEMORY_SCOPE_SYSTEM);
    if (__all(v >= thr)) break;
    if ((++spins & 31u) == 0) {
      const u64 now = __builtin_amdgcn_s_memrealtime();
      if (t0 == 0) t0 = now;
      if (__hip_atomic_load(tmo, __ATOMIC_RELAXED, __HIP_MEMORY_SCOPE_AGENT) == call ||
          now - t0 > DEADLINE_TICKS) {
        __hip_atomic_store(tmo, call, __ATOMIC_RELAXED, __HIP_MEMORY_SCOPE_AGENT);
        return false;
      }
    }
    __builtin_amdgcn_s_sleep(PSLEEP);
  }
  asm volatile("" ::: "memory");
  return true;
}

__device__ __forceinline__ bool wait_flag(const u32* f, u32 call, u32* tmo) { return wait_ge(f, call, call, tmo); }

template <int N> __device__ __forceinline__ void wait_vm() {
  asm volatile("s_waitcnt vmcnt(%0)" :: "i"(N) : "memory");
}

__device__ __forceinline__ u32 pack_bf16(float a, float b) {
  u32 r;
  asm("v_cvt_pk_bf16_f32 %0, %1, %2" : "=v"(r) : "v"(a), "v"(b));
  return r;
}
__device__ __forceinline__ float silu(float y) {
  const float yc = fminf(fmaxf(y, -60.0f), 60.0f);
  return y / (1.0f + __expf(-yc));
}

constexpr int PROW = 272;
constexpr int PART_BYTES = 32 * PROW;
constexpr int LDS_BYTES = 8 * PART_BYTES + 64;

#define ORDER 1
constexpr bool round_is_w(int n) { return ORDER ? (n == 0 || n == 2) : (n < 2); }
constexpr int round_base(int n) {
  return ORDER ? (n == 0 ? 0 : n == 1 ? 0 : n == 2 ? 8 : 8 * (n - 2)) : (n < 2 ? 8 * n : 8 * (n - 2));
}

__device__ __forceinline__ char* pick4(char* a, char* b, char* c, char* d, int sel) {
  char* lo = (sel & 1) ? b : a;
  char* hi = (sel & 1) ? d : c;
  return (sel & 2) ? hi : lo;
}

__global__ void __launch_bounds__(512, 2) gemm_ar_silu(Params p) {
  __shared__ __attribute__((aligned(16))) char smem[LDS_BYTES];
  const int tid = threadIdx.x, lane = tid & 63;
  const int wv = __builtin_amdgcn_readfirstlane(tid >> 6);
  const int bid = blockIdx.x;
  const int rank = p.rank;
  char* myws = p.ws[rank];
  u32* cntp = (u32*)(myws + CNT_OFF + (size_t)bid * FSTR);
  u32* tmo = (u32*)(myws + TMO_OFF);
  TS(0);

  if (bid < NPUSH) {
    const int dst = bid & 7, sub = bid >> 3;
    const int di = dst >> 2, dj = dst & 3;
    char* dws = p.ws[dst];
    const u32 cnt = __hip_atomic_load(cntp, __ATOMIC_RELAXED, __HIP_MEMORY_SCOPE_AGENT);
    u32* lcnt = (u32*)smem;
    u32 call = 0;
    if (wv >= 4) {
      if (wv == 4 && lane < 16) lcnt[lane] = 0u;
      asm volatile("s_waitcnt lgkmcnt(0)" ::: "memory");
      __builtin_amdgcn_s_barrier();
      asm volatile("" ::: "memory");
      call = __builtin_amdgcn_readfirstlane(cnt) + 1u;
    }
    if (wv == 4) {
#pragma unroll
      for (int n = 0; n < NT; ++n) {
        u32* fp; bool fa;
        const int ti = round_base(n) + sub;
        if (round_is_w(n)) {
          const int kh = ti & 1, g = ti >> 1;
          fp = (u32*)(dws + WF_OFF + (size_t)g * FSTR) + (rank * 2 + kh); fa = lane == 0;
        } else {
          const int rfi = 2 * ti + (lane & 1);
          fp = (u32*)(dws + XF_OFF + (size_t)(rfi >> 2) * FSTR) + ((rfi & 3) * 8 + rank); fa = lane < 2;
        }
        while (__hip_atomic_load(lcnt + n, __ATOMIC_RELAXED, __HIP_MEMORY_SCOPE_WORKGROUP) < 4u)
          __builtin_amdgcn_s_sleep(1);
        asm volatile("" ::: "memory");
        if (fa) st_flag(fp, call);
        TS(2 + n);
      }
    } else if (wv < 4) {
    const int c = lane & 15, kg = lane >> 4;
    u32 r[NT][16];
#pragma unroll
    for (int n = 0; n < NT; ++n) {
      const int ti = round_base(n) + sub;
      if (round_is_w(n)) {
        const int kh = ti & 1, g = ti >> 1, ks = 4 * kh + wv;
        const u16* src = p.w + (size_t)(32 * ks + 8 * kg) * NN + 512 * dj + 64 * g + 4 * c;
#pragma unroll
        for (int jj = 0; jj < 8; ++jj) {
          const u32x2 v = *(const u32x2*)(src + (size_t)jj * NN);
          r[n][2 * jj] = v.x; r[n][2 * jj + 1] = v.y;
        }
      } else {
        const int rfi = 2 * ti;
        const u16* src = p.x + (size_t)(1024 * di + 16 * rfi + c) * KSH + 64 * wv + 8 * kg;
#pragma unroll
        for (int k = 0; k < 4; ++k) {
          const u32x4 v = *(const u32x4*)(src + (k >> 1) * 16 * KSH + 32 * (k & 1));
          r[n][4 * k] = v.x; r[n][4 * k + 1] = v.y; r[n][4 * k + 2] = v.z; r[n][4 * k + 3] = v.w;
        }
      }
    }
    const __amdgpu_buffer_rsrc_t wrs = mk_rsrc(dws + WS_OFF + (size_t)rank * WS_PER, (u32)WS_PER);
    const __amdgpu_buffer_rsrc_t xrs = mk_rsrc(dws + XS_OFF + (size_t)rank * XS_PER, (u32)XS_PER);
    TS(1);
#pragma unroll
    for (int n = 0; n < NT; ++n) {
      if (round_is_w(n)) {
        u32 f[16];
#pragma unroll
        for (int d = 0; d < 4; ++d) {
          const u32 a0 = r[n][4 * d], a1 = r[n][4 * d + 1], b0 = r[n][4 * d + 2], b1 = r[n][4 * d + 3];
          f[0 + d] = __builtin_amdgcn_perm(b0, a0, 0x05040100u);
          f[4 + d] = __builtin_amdgcn_perm(b0, a0, 0x07060302u);
          f[8 + d] = __builtin_amdgcn_perm(b1, a1, 0x05040100u);
          f[12 + d] = __builtin_amdgcn_perm(b1, a1, 0x07060302u);
        }
#pragma unroll
        for (int e = 0; e < 16; ++e) r[n][e] = f[e];
      }
    }
    auto store_k = [&](int n, int k) {
      const u32x4 v = {r[n][4 * k], r[n][4 * k + 1], r[n][4 * k + 2], r[n][4 * k + 3]};
      const int ti = round_base(n) + sub;
      if (round_is_w(n)) {
        const int kh = ti & 1, g = ti >> 1, ks = 4 * kh + wv;
        st_sys(v, wrs, (u32)(((g * 8 + ks) * 4 + k) * 1024 + lane * 16));
      } else {
        const int rfi = 2 * ti + (k >> 1);
        st_sys(v, xrs, (u32)((rfi * 8 + 2 * wv + (k & 1)) * 1024 + lane * 16));
      }
    };
#pragma unroll
    for (int k = 0; k < 4; ++k) store_k(0, k);
    __builtin_amdgcn_s_barrier();
    asm volatile("" ::: "memory");
#pragma unroll
    for (int n = 0; n < NT; ++n) {
      if (n + 1 < NT) {
#pragma unroll
        for (int k = 0; k < LA; ++k) store_k(n + 1, k);
        wait_vm<LA>();
      } else {
        wait_vm<0>();
      }
      if (lane == 0) __hip_atomic_fetch_add(lcnt + n, 1u, __ATOMIC_RELAXED, __HIP_MEMORY_SCOPE_WORKGROUP);
      TS(2 + n);
      if (n + 1 < NT) {
#pragma unroll
        for (int k = LA; k < 4; ++k) store_k(n + 1, k);
      }
    }
    call = __builtin_amdgcn_readfirstlane(cnt) + 1u;
    }
    bool okp = true;
    if (dst != rank) {
      const int ri = rank >> 2, rj = rank & 3;
      const int sl = wv >= 5 ? wv - 5 : (wv == 4 ? 3 : wv + 4);
      const __amdgpu_buffer_rsrc_t src = mk_rsrc(p.out[rank], (u32)(MM * NN * 2));
      const __amdgpu_buffer_rsrc_t drs = mk_rsrc(p.out[dst], (u32)(MM * NN * 2));
#pragma unroll 1
      for (int m = ND / 8; m < 4; ++m) {
        const int i = sl * 8 + sub + 64 * m, T = i >> 2, r4 = i & 3;
        const int b = T < 16 ? (T & 7) : 8 + ((T - 16) >> 1);
        const int h = T < 16 ? (T >> 3) : (T & 1);
        okp = wait_flag((const u32*)(myws + TF_OFF + (size_t)b * FSTR) + 4 * h + (lane & 3), call, tmo) && okp;
        const u32 off = (u32)(((1024 * ri + 32 * b + 8 * r4 + (lane >> 5)) * NN + 512 * rj) * 2 + 512 * h + (lane & 31) * 16);
        u32x4 v[4];
#pragma unroll
        for (int k = 0; k < 4; ++k) v[k] = __builtin_amdgcn_raw_buffer_load_b128(src, off + k * 2 * NN * 2, 0, AUX_SYS);
#pragma unroll
        for (int k = 0; k < 4; ++k) {
          if (!okp) v[k] = u32x4{0x7fc07fc0u, 0x7fc07fc0u, 0x7fc07fc0u, 0x7fc07fc0u};
          st_sys(v[k], drs, off + k * 2 * NN * 2);
        }
        TS(10 + m);
      }
    }
    wait_vm<0>();
    TS(14);
    u32 old = 0;
    if (lane == 0) old = __hip_atomic_fetch_add(lcnt + 15, 1u, __ATOMIC_RELAXED, __HIP_MEMORY_SCOPE_WORKGROUP);
    old = __builtin_amdgcn_readfirstlane(old);
    if (old == 7u) {
      if (dst != rank) {
        if (lane == 0) st_flag((u32*)(dws + DF_OFF + (size_t)sub * FSTR) + rank, call);
        const u32* wf = (ND && lane >= 1 && lane <= 8) ? (const u32*)(myws + OF_OFF + (size_t)(8 * sub + lane - 1) * FSTR) + dst
                                                       : (const u32*)(myws + DF_OFF + (size_t)sub * FSTR) + dst;
        const bool ok2 = wait_flag(wf, call, tmo);
        if (!ok2 && lane == 0) {
          ((u16*)p.out[rank])[(size_t)(1024 * (rank >> 2)) * NN + 512 * (rank & 3)] = 0x7fc0;
        }
      }
      if (lane == 0) __hip_atomic_store(cntp, call, __ATOMIC_RELAXED, __HIP_MEMORY_SCOPE_AGENT);
    }
    TS(15);
    TS_FLUSH();
    return;
  }

  const int gid = bid - NPUSH;
  const int g = gid & 7, b0 = gid >> 3;
  const int ri = rank >> 2, rj = rank & 3;
  const u32 cnt = __hip_atomic_load(cntp, __ATOMIC_RELAXED, __HIP_MEMORY_SCOPE_AGENT);
  const u32 call = __builtin_amdgcn_readfirstlane(cnt) + 1u;
  const int q = wv;
  const __amdgpu_buffer_rsrc_t wr = mk_rsrc(myws + WS_OFF + (size_t)q * WS_PER, (u32)WS_PER);
  const __amdgpu_buffer_rsrc_t xr = mk_rsrc(myws + XS_OFF + (size_t)q * XS_PER, (u32)XS_PER);
  const __amdgpu_buffer_rsrc_t ors = mk_rsrc(p.out[rank], (u32)(MM * NN * 2));
  const u32* wflag = (const u32*)(myws + WF_OFF + (size_t)g * FSTR) + q * 2 + (lane & 1);
  auto xflag = [&](int b) {
    return (const u32*)(myws + XF_OFF + (size_t)(b >> 1) * FSTR) + (b & 1) * 16 + (lane & 1) * 8 + q;
  };
  u32* sbad = (u32*)(smem + 8 * PART_BYTES);

  u32x4 wf[8][4];
  u32x4 xf[2][8];
  auto load_w = [&]() {
#pragma unroll
    for (int s = 0; s < 8; ++s)
#pragma unroll
      for (int f = 0; f < 4; ++f)
        wf[s][f] = ld_sys(wr, (u32)(((g * 8 + s) * 4 + f) * 1024 + lane * 16));
    __builtin_amdgcn_sched_barrier(0);
  };
  auto load_x = [&](int b) {
#pragma unroll
    for (int s = 0; s < 8; ++s)
#pragma unroll
      for (int rf = 0; rf < 2; ++rf)
        xf[rf][s] = ld_sys(xr, (u32)(((b * 2 + rf) * 8 + s) * 1024 + lane * 16));
    __builtin_amdgcn_sched_barrier(0);
  };
  bool ok;
  auto tile = [&](int b, int tsb) {
    f32x4 acc[2][4];
#pragma unroll
    for (int rf = 0; rf < 2; ++rf)
#pragma unroll
      for (int f = 0; f < 4; ++f) acc[rf][f] = f32x4{0.f, 0.f, 0.f, 0.f};
#pragma unroll
    for (int s = 0; s < 8; ++s)
#pragma unroll
      for (int rf = 0; rf < 2; ++rf)
#pragma unroll
        for (int f = 0; f < 4; ++f)
          acc[rf][f] = __builtin_amdgcn_mfma_f32_16x16x32_bf16(
              __builtin_bit_cast(bf16x8, wf[s][f]), __builtin_bit_cast(bf16x8, xf[rf][s]), acc[rf][f], 0, 0, 0);
    TS(tsb);
    {
      char* part = smem + wv * PART_BYTES;
      const int m = lane & 15, q4 = lane >> 4;
#pragma unroll
      for (int rf = 0; rf < 2; ++rf)
#pragma unroll
        for (int reg = 0; reg < 4; ++reg) {
          const f32x4 v = {acc[rf][0][reg], acc[rf][1][reg], acc[rf][2][reg], acc[rf][3][reg]};
          *(f32x4*)(part + (16 * rf + m) * PROW + q4 * 64 + reg * 16) = v;
        }
      if (lane == 0) sbad[wv] = ok ? 0u : 1u;
    }
    asm volatile("s_waitcnt lgkmcnt(0)" ::: "memory");
    __builtin_amdgcn_s_barrier();
    asm volatile("" ::: "memory");
    TS(tsb + 1);
    {
      const int orow = tid >> 4, oseg = tid & 15;
      f32x4 pv[8];
#pragma unroll
      for (int k = 0; k < 8; ++k) pv[k] = *(const f32x4*)(smem + k * PART_BYTES + orow * PROW + oseg * 16);
      const f32x4 s0 = ((pv[0] + pv[1]) + (pv[2] + pv[3])) + ((pv[4] + pv[5]) + (pv[6] + pv[7]));
      const u32x4 bad0 = *(const u32x4*)sbad, bad1 = *(const u32x4*)(sbad + 4);
      const bool bad = (bad0.x | bad0.y | bad0.z | bad0.w | bad1.x | bad1.y | bad1.z | bad1.w) != 0u;
      u32x2 o;
      o.x = pack_bf16(silu(s0[0]), silu(s0[1]));
      o.y = pack_bf16(silu(s0[2]), silu(s0[3]));
      if (bad) o = u32x2{0x7fc07fc0u, 0x7fc07fc0u};
      const u32 ooff = (u32)(((1024 * ri + 32 * b + orow) * NN + 512 * rj + 64 * g + 4 * oseg) * 2);
      __builtin_amdgcn_raw_buffer_store_b64(o, ors, ooff, 0, AUX_SYS);
    }
    wait_vm<0>();
    __builtin_amdgcn_s_barrier();
    asm volatile("" ::: "memory");
    TS(tsb + 2);
    if (tid == 0) st_flag((u32*)(myws + TF_OFF + (size_t)b * FSTR) + g, call);
  };

  if (ORDER == 1 && b0 < 8 && g >= 4) {
    ok = wait_flag(xflag(b0), call, tmo);
    TS(1);
    load_x(b0);
    ok = wait_flag(wflag, call, tmo) && ok;
    TS(2);
    load_w();
  } else {
    ok = wait_flag(wflag, call, tmo);
    TS(1);
    load_w();
    ok = wait_flag(xflag(b0), call, tmo) && ok;
    TS(2);
    load_x(b0);
  }
  tile(b0, 3);
  ok = wait_flag(xflag(b0 + 16), call, tmo) && ok;
  TS(6);
  load_x(b0 + 16);
  tile(b0 + 16, 7);
  if (tid == 0) __hip_atomic_store(cntp, call, __ATOMIC_RELAXED, __HIP_MEMORY_SCOPE_AGENT);
  TS_FLUSH();
}

extern "C" void kernel_launch(int rank, int nranks, ncclComm_t comm,
                              hipStream_t stream,
                              void* const* d_in, const int* in_sizes, int n_in,
                              void* d_out, int out_size,
                              const int* in_dtypes, int out_dtype,
                              void* const* d_out_all,
                              void* const* d_ws_all, size_t ws_bytes) {
  (void)comm; (void)in_sizes; (void)n_in; (void)d_out; (void)out_size; (void)in_dtypes; (void)out_dtype;
  if (nranks != NR || (unsigned)rank >= (unsigned)NR || ws_bytes < WS_NEEDED) {
    fprintf(stderr, "unsupported configuration\n");
    return;
  }
  Params p;
  p.x = (const u16*)d_in[0];
  p.w = (const u16*)d_in[1];
  for (int i = 0; i < NR; ++i) { p.out[i] = (char*)d_out_all[i]; p.ws[i] = (char*)d_ws_all[i]; }
  p.rank = rank;
  gemm_ar_silu<<<GRID, 512, 0, stream>>>(p);
  HIP_CHECK(hipGetLastError());
}
```

```cpp
#include <hip/hip_runtime.h>
#include <rccl/rccl.h>
#include <cstdio>
#include <cstdint>

#define HIP_CHECK(expr) do { hipError_t _e = (expr); if (_e != hipSuccess) { \
    fprintf(stderr, "HIP error %s at %s:%d\n", hipGetErrorString(_e), __FILE__, __LINE__); \
    return; } } while (0)

typedef unsigned int u32;
typedef unsigned long long u64;
typedef unsigned short u16;
typedef __attribute__((ext_vector_type(4))) unsigned int u32x4;
typedef __attribute__((ext_vector_type(2))) unsigned int u32x2;
typedef __attribute__((ext_vector_type(8))) short bf16x8;
typedef __attribute__((ext_vector_type(4))) float f32x4;

#define PSLEEP 2
#define DEPTH 3
static_assert(DEPTH >= 2 && DEPTH <= 4, "");
#define DEPTH2 3
#define MSW 4
static_assert(DEPTH2 >= 2 && DEPTH2 <= 4, "");
#define ESLEEP 1
#define DSLEEP 4
constexpr int NR = 8;
constexpr int MM = 2048, NN = 2048, KSH = 256;
constexpr int PPD = 8;
constexpr int NPUSH = PPD * NR;
constexpr int NT = 6;
constexpr int NGEMM = 128;
#define SPLIT 0
constexpr int GRID = NPUSH + NGEMM;
static_assert(GRID <= 256, "all workgroups must be co-resident");

constexpr size_t XS_PER = 512u << 10;
constexpr size_t WS_PER = 256u << 10;
constexpr size_t XS_OFF = 0;
constexpr size_t WS_OFF = NR * XS_PER;
constexpr size_t SLOT_BYTES = NR * (XS_PER + WS_PER);
constexpr size_t FLAG_OFF = SLOT_BYTES;
constexpr size_t FSTR = 128;
constexpr size_t WF_OFF = FLAG_OFF;
constexpr size_t XF_OFF = WF_OFF + 8 * FSTR;
constexpr size_t TF_OFF = XF_OFF + 16 * FSTR;
constexpr size_t DF_OFF = TF_OFF + 32 * FSTR;
constexpr size_t AF_OFF = DF_OFF + 32 * FSTR;
constexpr size_t CNT_OFF = AF_OFF + 8 * FSTR;
constexpr size_t TMO_OFF = CNT_OFF + (size_t)GRID * FSTR;
constexpr size_t WS_NEEDED = TMO_OFF + FSTR;

constexpr u64 DEADLINE_TICKS = 40000000ull;

#define TS(i) do {} while (0)
#define TS_FLUSH() do {} while (0)
#define TS_ARG
#define TS_PASS

struct Params {
  const u16* x;
  const u16* w;
  char* myws;
  char* myout;
  int rank;
  int pad_[3];
  char* ws[NR];
  char* out[NR];
};

constexpr int AUX_SYS = 17;
#define AUX_LD 17

__device__ __forceinline__ __amdgpu_buffer_rsrc_t mk_rsrc(const void* p, u32 bytes) {
  return __builtin_amdgcn_make_buffer_rsrc((void*)p, 0, bytes, 0x00020000);
}
__device__ __forceinline__ void st_sys(u32x4 v, __amdgpu_buffer_rsrc_t r, u32 off) {
  __builtin_amdgcn_raw_buffer_store_b128(v, r, off, 0, AUX_SYS);
}
__device__ __forceinline__ u32x4 ld_sys(__amdgpu_buffer_rsrc_t r, u32 off) {
  return __builtin_amdgcn_raw_buffer_load_b128(r, off, 0, AUX_LD);
}

typedef __attribute__((address_space(1))) u32 gu32;
__device__ __forceinline__ void st_flag(u32* f, u32 v) {
  __hip_atomic_store((gu32*)(size_t)f, v, __ATOMIC_RELAXED, __HIP_MEMORY_SCOPE_SYSTEM);
}

__device__ __forceinline__ bool wait_ge(const u32* f, u32 thr, u32 call, u32* tmo) {
  u64 t0 = 0;
  u32 spins = 0;
  for (;;) {
    const u32 v = __hip_atomic_load(f, __ATOMIC_RELAXED, __HIP_MEMORY_SCOPE_SYSTEM);
    if (__all(v >= thr)) break;
    if ((++spins & 31u) == 0) {
      const u64 now = __builtin_amdgcn_s_memrealtime();
      if (t0 == 0) t0 = now;
      if (__hip_atomic_load(tmo, __ATOMIC_RELAXED, __HIP_MEMORY_SCOPE_AGENT) == call ||
          now - t0 > DEADLINE_TICKS) {
        __hip_atomic_store(tmo, call, __ATOMIC_RELAXED, __HIP_MEMORY_SCOPE_AGENT);
        return false;
      }
    }
    __builtin_amdgcn_s_sleep(PSLEEP);
  }
  asm volatile("" ::: "memory");
  return true;
}

__device__ __forceinline__ bool wait_flag(const u32* f, u32 call, u32* tmo) { return wait_ge(f, call, call, tmo); }

template <int N> __device__ __forceinline__ void wait_vm() {
  asm volatile("s_waitcnt vmcnt(%0)" :: "i"(N) : "memory");
}

__device__ __forceinline__ u32 pack_bf16(float a, float b) {
  u32 r;
  asm("v_cvt_pk_bf16_f32 %0, %1, %2" : "=v"(r) : "v"(a), "v"(b));
  return r;
}
__device__ __forceinline__ float silu(float y) {
  const float yc = fminf(fmaxf(y, -60.0f), 60.0f);
  return y / (1.0f + __expf(-yc));
}

constexpr int PROW = 272;
constexpr int PART_BYTES = 16 * PROW;
constexpr int LDS_BYTES = 8 * PART_BYTES + 64;

constexpr bool round_is_w(int n) { return n < 2; }

constexpr int pace_cnt(int j) {
  int x = DEPTH - 1;
  for (int n = 0; n + 2 < NT; ++n)
    if (j - DEPTH <= 4 * n + 3 && 4 * n + 3 <= j - 1) x += 4;
  return x;
}
template <int I> struct IC { static constexpr int value = I; };
template <int N, int I = 0, class F> __device__ __forceinline__ void static_for(F&& f) {
  if constexpr (I < N) { f(IC<I>{}); static_for<N, I + 1>(f); }
}

__device__ __forceinline__ char* pick4(char* a, char* b, char* c, char* d, int sel) {
  char* lo = (sel & 1) ? b : a;
  char* hi = (sel & 1) ? d : c;
  return (sel & 2) ? hi : lo;
}

template <int D>
__device__ __forceinline__ void store4(int sel, u32x4 (&v)[4], u32 vo, u32x4 rs, u32 s0, u32 s1, u32 s2, u32 s3) {
  constexpr int E = D - 1 < 4 ? D - 1 : 4;
  asm volatile("s_cmp_lt_u32 %10, 2\n\t"
               "s_cbranch_scc1 .Lnp_%=\n\t"
               "s_cmp_eq_u32 %10, 3\n\t"
               "s_cbranch_scc1 .Lw3_%=\n\t"
               "s_waitcnt vmcnt(4)\n"
               ".Lw3_%=:\n\t"
               "s_waitcnt vmcnt(%13)\n\t"
               "buffer_store_dwordx4 %0, %4, %5, %6 offen sc0 sc1\n\t"
               "s_waitcnt vmcnt(%14)\n\t"
               "buffer_store_dwordx4 %1, %4, %5, %7 offen sc0 sc1\n\t"
               "s_waitcnt vmcnt(%15)\n\t"
               "buffer_store_dwordx4 %2, %4, %5, %8 offen sc0 sc1\n\t"
               "s_waitcnt vmcnt(%16)\n\t"
               "s_branch .Ltail_%=\n"
               ".Lnp_%=:\n\t"
               "s_cmp_eq_u32 %10, 1\n\t"
               "s_cbranch_scc1 .Lw1_%=\n\t"
               "s_waitcnt vmcnt(0)\n"
               ".Lw1_%=:\n\t"
               "s_waitcnt vmcnt(%12)\n\t"
               "buffer_store_dwordx4 %0, %4, %5, %6 offen sc0 sc1\n\t"
               "s_waitcnt vmcnt(%11)\n\t"
               "buffer_store_dwordx4 %1, %4, %5, %7 offen sc0 sc1\n\t"
               "s_waitcnt vmcnt(%11)\n\t"
               "buffer_store_dwordx4 %2, %4, %5, %8 offen sc0 sc1\n\t"
               "s_waitcnt vmcnt(%11)\n"
               ".Ltail_%=:\n\t"
               "buffer_store_dwordx4 %3, %4, %5, %9 offen sc0 sc1\n\t"
               "s_nop 1"
               :: "v"(v[0]), "v"(v[1]), "v"(v[2]), "v"(v[3]),
                  "v"(vo), "s"(rs), "s"(s0), "s"(s1), "s"(s2), "s"(s3),
                  "s"(sel), "i"(D - 1), "i"(E), "i"(E + 4),
                  "i"(1 <= D - 1 ? D + 3 : D - 1), "i"(2 <= D - 1 ? D + 3 : D - 1), "i"(3 <= D - 1 ? D + 3 : D - 1)
               : "scc", "memory");
}

constexpr u32 SENT = 0xffffffffu;
template <int N>
__device__ __forceinline__ bool valid4(u32x4 (&v)[4]) {
  u64 m; u32 t;
  asm volatile("s_waitcnt vmcnt(%18)\n\t"
               "v_max3_u32 %1, %2, %3, %4\n\t"
               "v_max3_u32 %1, %1, %5, %6\n\t"
               "v_max3_u32 %1, %1, %7, %8\n\t"
               "v_max3_u32 %1, %1, %9, %10\n\t"
               "v_max3_u32 %1, %1, %11, %12\n\t"
               "v_max3_u32 %1, %1, %13, %14\n\t"
               "v_max3_u32 %1, %1, %15, %16\n\t"
               "v_max_u32 %1, %1, %17\n\t"
               "v_cmp_eq_u32_e64 %0, -1, %1\n\t"
               "s_nop 4"
               : "=s"(m), "=&v"(t)
               : "v"(v[0].x), "v"(v[0].y), "v"(v[0].z), "v"(v[0].w), "v"(v[1].x), "v"(v[1].y), "v"(v[1].z), "v"(v[1].w),
                 "v"(v[2].x), "v"(v[2].y), "v"(v[2].z), "v"(v[2].w), "v"(v[3].x), "v"(v[3].y), "v"(v[3].z), "v"(v[3].w),
                 "i"(N)
               : "memory");
  return m == 0ull;
}

__global__ void __launch_bounds__(512, 2) gemm_ar_silu(Params p) {
  __shared__ __attribute__((aligned(16))) char smem[LDS_BYTES];
  const int tid = threadIdx.x, lane = tid & 63;
  const int wv = __builtin_amdgcn_readfirstlane(tid >> 6);
  const int bid = blockIdx.x;
  const bool is_engine = bid < NPUSH;
  const int dst = bid & 7;
  const int sub = bid >> 3;
#define GMAP 2
  const int gx = bid & 7, gs = (bid - NPUSH) >> 3;
  const int g = ((gx & ((8 >> GMAP) - 1)) << GMAP) + (gs & ((1 << GMAP) - 1));
  const int r16 = (gx >> (3 - GMAP)) * (16 >> GMAP) + (gs >> GMAP);
  int rank = p.rank;
  u64 a0 = (u64)p.myws, a1 = (u64)p.myout, a2 = (u64)p.x, a3 = (u64)p.w, a4 = (u64)p.ws[dst], a5 = (u64)p.out[dst];
  asm volatile("" : "+s"(rank), "+s"(a0), "+s"(a1), "+s"(a2), "+s"(a3), "+s"(a4), "+s"(a5));
  typedef __attribute__((address_space(1))) char gchar;
  char* const myws = (char*)(gchar*)a0;
  char* const myout = (char*)(gchar*)a1;
  const u16* const px = (const u16*)(char*)(gchar*)a2;
  const u16* const pw = (const u16*)(char*)(gchar*)a3;
  char* const dws = (char*)(gchar*)a4;
  char* const dout = (char*)(gchar*)a5;
  u32* cntp = (u32*)(myws + CNT_OFF + (size_t)bid * FSTR);
  u32* tmo = (u32*)(myws + TMO_OFF);
  TS(0);

  if (__builtin_expect(is_engine, 1)) {
    if (wv > 5) return;
    const int di = dst >> 2, dj = dst & 3;
    u32* lcnt = (u32*)smem;
    if (wv >= 4) {
      const u32 cnt = __hip_atomic_load(cntp, __ATOMIC_RELAXED, __HIP_MEMORY_SCOPE_AGENT);
      __builtin_amdgcn_s_setprio(3);
      const u32 call = __builtin_amdgcn_readfirstlane(cnt) + 1u;
      if (wv == 4 && lane < 17) lcnt[lane] = lane == 16 ? call : 0u;
      asm volatile("s_waitcnt lgkmcnt(0)" :: "s"(call) : "memory");
      __builtin_amdgcn_s_barrier();
      asm volatile("" ::: "memory");
      if (wv >= 5) {
        if (dst == rank) return;
        {
          const __amdgpu_buffer_rsrc_t mrs = mk_rsrc(myout, (u32)(MM * NN * 2));
          const u32 eo = (u32)(((1024 * di + 16 * (49 + 2 * sub)) * NN + 512 * dj) * 2 + lane * 16);
#pragma unroll
          for (int k = 0; k < 16; ++k) st_sys(u32x4{SENT, SENT, SENT, SENT}, mrs, eo + k * NN * 2);
          wait_vm<0>();
          if (lane == 0) __hip_atomic_store(lcnt + 14, 1u, __ATOMIC_RELAXED, __HIP_MEMORY_SCOPE_WORKGROUP);
        }
        if (wait_flag((const u32*)(myws + AF_OFF + (size_t)sub * FSTR) + (lane & 31), call, tmo) && lane == 0)
          __hip_atomic_fetch_or(lcnt + 12, 0x100u, __ATOMIC_RELAXED, __HIP_MEMORY_SCOPE_WORKGROUP);
        while (__hip_atomic_load(lcnt + 4, __ATOMIC_RELAXED, __HIP_MEMORY_SCOPE_WORKGROUP) < 4u)
          __builtin_amdgcn_s_sleep(4);
        const u32* tf = (const u32*)(myws + TF_OFF) + (16 * (lane >> 4) + 2 * sub + ((lane >> 3) & 1)) * 8 + (lane & 7);
        u64 t0 = 0; u32 spins = 0, prev = 0;
        for (;;) {
          const u32 v = __hip_atomic_load(tf, __ATOMIC_RELAXED, __HIP_MEMORY_SCOPE_SYSTEM);
          const u64 bm = __ballot(v >= call);
          u32 rm = 0;
#pragma unroll
          for (int m = 0; m < 8; ++m) rm |= (((bm >> (8 * m)) & 0xffull) == 0xffull) ? (1u << m) : 0u;
          if (rm != prev) {
            if (lane == 0) __hip_atomic_fetch_or(lcnt + 12, rm, __ATOMIC_RELAXED, __HIP_MEMORY_SCOPE_WORKGROUP);
            prev = rm;
          }
          if (rm == 0xffu) break;
          if ((++spins & 31u) == 0) {
            const u64 now = __builtin_amdgcn_s_memrealtime();
            if (t0 == 0) t0 = now;
            if (__hip_atomic_load(tmo, __ATOMIC_RELAXED, __HIP_MEMORY_SCOPE_AGENT) == call || now - t0 > DEADLINE_TICKS) {
              __hip_atomic_store(tmo, call, __ATOMIC_RELAXED, __HIP_MEMORY_SCOPE_AGENT);
              if (lane == 0) {
                __hip_atomic_store(lcnt + 13, 1u, __ATOMIC_RELAXED, __HIP_MEMORY_SCOPE_WORKGROUP);
                asm volatile("s_waitcnt lgkmcnt(0)" ::: "memory");
                __hip_atomic_fetch_or(lcnt + 12, 0xffu, __ATOMIC_RELAXED, __HIP_MEMORY_SCOPE_WORKGROUP);
              }
              break;
            }
          }
          __builtin_amdgcn_s_sleep(PSLEEP);
        }
        TS(10);
        TS_FLUSH();
        return;
      }
#pragma unroll
      for (int h = 0; h < 2 * NT; ++h) {
        u32* fp;
        if (h < 4) {
          fp = (u32*)(dws + WF_OFF + (size_t)sub * FSTR) + (rank * 4 + h);
        } else {
          const int rfi = 2 * (8 * ((h - 4) >> 1) + sub) + (h & 1);
          fp = (u32*)(dws + XF_OFF + (size_t)(rfi >> 2) * FSTR) + ((rfi & 3) * 8 + rank);
        }
        while (__hip_atomic_load(lcnt + h, __ATOMIC_RELAXED, __HIP_MEMORY_SCOPE_WORKGROUP) < 4u)
          __builtin_amdgcn_s_sleep(1);
        if (h == 3 && dst != rank)
          while (__hip_atomic_load(lcnt + 14, __ATOMIC_RELAXED, __HIP_MEMORY_SCOPE_WORKGROUP) == 0u)
            __builtin_amdgcn_s_sleep(1);
        asm volatile("" ::: "memory");
        if (lane == 0) st_flag(fp, call);
        if (h & 1) TS(2 + (h >> 1));
      }
      if (dst != rank) {
        while (__hip_atomic_load(lcnt + 15, __ATOMIC_RELAXED, __HIP_MEMORY_SCOPE_WORKGROUP) < 4u)
          __builtin_amdgcn_s_sleep(1);
        asm volatile("" ::: "memory");
        TS(14);
        if (lane == 0) st_flag((u32*)(dws + DF_OFF + (size_t)sub * FSTR) + rank, call);
        const bool ok2 = wait_flag((const u32*)(myws + DF_OFF + (size_t)sub * FSTR) + dst, call, tmo);
        if (!ok2 && lane == 0) {
          ((u16*)myout)[(size_t)(1024 * (rank >> 2)) * NN + 512 * (rank & 3)] = 0x7fc0;
        }
      }
      if (lane == 0) __hip_atomic_store(cntp, call, __ATOMIC_RELAXED, __HIP_MEMORY_SCOPE_AGENT);
      TS(15);
      TS_FLUSH();
      return;
    }
    const int c = lane & 15, kg = lane >> 4;
    u32 r[NT][16];
    auto load_task = [&](int n) {
      if (round_is_w(n)) {
        const int kh = n, g = sub, ks = 4 * kh + wv;
        const u16* src = pw + (size_t)(32 * ks + 8 * kg) * NN + 512 * dj + 64 * g + 4 * c;
#pragma unroll
        for (int jj = 0; jj < 8; ++jj) {
          const u32x2 v = *(const u32x2*)(src + (size_t)jj * NN);
          r[n][2 * jj] = v.x; r[n][2 * jj + 1] = v.y;
        }
      } else {
        const int rfi = 2 * (8 * (n - 2) + sub);
        const u16* src = px + (size_t)(1024 * di + 16 * rfi + c) * KSH + 64 * wv + 8 * kg;
#pragma unroll
        for (int k = 0; k < 4; ++k) {
          const u32x4 v = *(const u32x4*)(src + (k >> 1) * 16 * KSH + 32 * (k & 1));
          r[n][4 * k] = v.x; r[n][4 * k + 1] = v.y; r[n][4 * k + 2] = v.z; r[n][4 * k + 3] = v.w;
        }
      }
    };
    auto transpose = [&](int n) {
      u32 f[16];
#pragma unroll
      for (int d = 0; d < 4; ++d) {
        const u32 a0 = r[n][4 * d], a1 = r[n][4 * d + 1], b0 = r[n][4 * d + 2], b1 = r[n][4 * d + 3];
        f[0 + d] = __builtin_amdgcn_perm(b0, a0, 0x05040100u);
        f[4 + d] = __builtin_amdgcn_perm(b0, a0, 0x07060302u);
        f[8 + d] = __builtin_amdgcn_perm(b1, a1, 0x05040100u);
        f[12 + d] = __builtin_amdgcn_perm(b1, a1, 0x07060302u);
      }
#pragma unroll
      for (int e = 0; e < 16; ++e) r[n][e] = f[e];
    };
    const __amdgpu_buffer_rsrc_t wrs = mk_rsrc(dws + WS_OFF + (size_t)rank * WS_PER, (u32)WS_PER);
    const __amdgpu_buffer_rsrc_t xrs = mk_rsrc(dws + XS_OFF + (size_t)rank * XS_PER, (u32)XS_PER);
    auto store_k = [&](int n, int k) {
      const u32x4 v = {r[n][4 * k], r[n][4 * k + 1], r[n][4 * k + 2], r[n][4 * k + 3]};
      if (round_is_w(n)) {
        const int kh = n, g = sub, ks = 4 * kh + wv;
        st_sys(v, wrs, (u32)(((g * 8 + ks) * 4 + k) * 1024 + lane * 16));
      } else {
        const int rfi = 2 * (8 * (n - 2) + sub) + (k >> 1);
        st_sys(v, xrs, (u32)((rfi * 8 + 2 * wv + (k & 1)) * 1024 + lane * 16));
      }
    };
    auto signal = [&](int n, int cnt = 1) {
      if (lane < cnt) __hip_atomic_fetch_add(lcnt + n + lane, 1u, __ATOMIC_RELAXED, __HIP_MEMORY_SCOPE_WORKGROUP);
    };
    constexpr int HS = (4 * (NT - 1) - DEPTH) / 2;
    load_task(0);
    __builtin_amdgcn_sched_barrier(0);
    load_task(1);
    __builtin_amdgcn_sched_barrier(0);
    TS(1);
    transpose(0);
    static_for<4 * (NT - 1)>([&](auto J) {
      constexpr int j = decltype(J)::value;
      if (j == DEPTH) {
        transpose(1);
        __builtin_amdgcn_s_barrier();
        asm volatile("" ::: "memory");
      }
      if (j >= DEPTH) {
        wait_vm<pace_cnt(j)>();
        if (((j - DEPTH) & 1) == 1) signal((j - DEPTH) >> 1);
        if (((j - DEPTH) & 3) == 3) TS(2 + ((j - DEPTH) >> 2));
      }
      store_k(j >> 2, j & 3);
      __builtin_amdgcn_sched_barrier(0);
      if ((j & 3) == 3 && (j >> 2) + 2 < NT) { load_task((j >> 2) + 2); __builtin_amdgcn_sched_barrier(0); }
    });
    const int ri = rank >> 2, rj = rank & 3;
    const u64 a6 = a4 + XS_OFF + (u64)rank * XS_PER;
    const u32x4 xrs4 = {(u32)a6, (u32)(a6 >> 32) & 0xffffu, (u32)XS_PER, 0x00020000u};
    const u32x4 srs = {(u32)a1, (u32)(a1 >> 32) & 0xffffu, (u32)(MM * NN * 2), 0x00020000u};
    const u32x4 drs = {(u32)a5, (u32)(a5 >> 32) & 0xffffu, (u32)(MM * NN * 2), 0x00020000u};
    const u32 off0 = (u32)(((1024 * ri + 32 * sub + 4 * wv) * NN + 512 * rj) * 2 + lane * 16);
    auto moff = [&](int m) -> u32 { return (u32)(16 * (m >> 1) + (m & 1)) * (u32)(16 * NN * 2); };
    auto ready = [&](int m) -> bool {
      return (__hip_atomic_load(lcnt + 12, __ATOMIC_RELAXED, __HIP_MEMORY_SCOPE_WORKGROUP) >> m) & 1u;
    };
    auto spin = [&](int m) { while (!ready(m)) __builtin_amdgcn_s_sleep(1); asm volatile("" ::: "memory"); };
    auto ld4 = [&](int m, u32x4 (&v)[4]) {
      const u32 so = moff(m);
      asm volatile("buffer_load_dwordx4 %0, %4, %5, %6 offen sc0 sc1\n\t"
                   "buffer_load_dwordx4 %1, %4, %5, %7 offen sc0 sc1\n\t"
                   "buffer_load_dwordx4 %2, %4, %5, %8 offen sc0 sc1\n\t"
                   "buffer_load_dwordx4 %3, %4, %5, %9 offen sc0 sc1"
                   : "=&v"(v[0]), "=&v"(v[1]), "=&v"(v[2]), "=&v"(v[3])
                   : "v"(off0), "s"(srs), "s"(so), "s"(so + NN * 2), "s"(so + 2 * NN * 2), "s"(so + 3 * NN * 2)
                   : "memory");
    };
    u32x4 A[4], B[4];
    bool early;
    {
      u32x4 X[4];
#pragma unroll
      for (int k = 0; k < 4; ++k) X[k] = u32x4{r[NT - 1][4 * k], r[NT - 1][4 * k + 1], r[NT - 1][4 * k + 2], r[NT - 1][4 * k + 3]};
      asm volatile("" : "+v"(X[0]), "+v"(X[1]), "+v"(X[2]), "+v"(X[3]) :: "memory");
      early = dst != rank && ready(8);
      if (early) { asm volatile("" ::: "memory"); ld4(0, A); }
      const u32 xo = (u32)((2 * (8 * (NT - 3) + sub) * 8 + 2 * wv) * 1024 + lane * 16);
      store4<DEPTH>(__builtin_amdgcn_readfirstlane(early ? 3 : 1), X, xo, xrs4, 0u, 1024u, 8192u, 9216u);
      signal(HS, 2 * NT - 2 - HS);
      TS(2 + NT - 2);
      if (early) early = valid4<4>(A);
      if (dst == rank) {
        wait_vm<0>();
        signal(2 * NT - 2, 2);
        TS(2 + NT - 1);
      }
    }
    if (dst != rank) {
      const bool late5 = early;
      if (!early) {
        bool first = true;
        for (;;) {
          u32 st = __hip_atomic_load(lcnt + 12, __ATOMIC_RELAXED, __HIP_MEMORY_SCOPE_WORKGROUP);
          if (st & 0x101u) {
            asm volatile("" ::: "memory");
            ld4(0, A);
          }
          const bool okv = valid4<0>(A);
          if (first) { signal(2 * NT - 2, 2); TS(2 + NT - 1); first = false; }
          if ((st & 1u) || ((st & 0x100u) && okv)) break;
          __builtin_amdgcn_s_sleep(DSLEEP);
        }
      }
      TS(8);
      auto step = [&](auto DD, int m, u32x4 (&cur)[4], u32x4 (&nxt)[4]) {
        const bool more = m + 1 < 8;
        const bool pre = more && ready(m + 1);
        if (pre) { asm volatile("" ::: "memory"); ld4(m + 1, nxt); }
        const u32 so = moff(m);
        store4<decltype(DD)::value>(__builtin_amdgcn_readfirstlane((early ? 1 : 0) + (pre ? 2 : 0)), cur, off0, drs,
                                    so, so + NN * 2, so + 2 * NN * 2, so + 3 * NN * 2);
        if (m == 0 && late5) { signal(2 * NT - 2, 2); TS(2 + NT - 1); }
        if (more && !pre) { spin(m + 1); ld4(m + 1, nxt); }
        early = pre;
      };
#pragma unroll 1
      for (int m = 0; m < MSW; m += 2) { step(IC<DEPTH>{}, m, A, B); step(IC<DEPTH>{}, m + 1, B, A); }
#pragma unroll 1
      for (int m = MSW; m < 8; m += 2) { step(IC<DEPTH2>{}, m, A, B); step(IC<DEPTH2>{}, m + 1, B, A); }
      if (__hip_atomic_load(lcnt + 13, __ATOMIC_RELAXED, __HIP_MEMORY_SCOPE_WORKGROUP) != 0u) {
        wait_vm<0>();
        const __amdgpu_buffer_rsrc_t prs = mk_rsrc(dout, (u32)(MM * NN * 2));
        for (int m = 0; m < 8; ++m)
          for (int k = 0; k < 4; ++k)
            st_sys(u32x4{0x7fc07fc0u, 0x7fc07fc0u, 0x7fc07fc0u, 0x7fc07fc0u}, prs, off0 + moff(m) + k * NN * 2);
        wait_vm<0>();
      }
      TS(13);
      signal(15);
      const u32 call = __builtin_amdgcn_readfirstlane(lcnt[16]);
      const __amdgpu_buffer_rsrc_t mrs = mk_rsrc(myout, (u32)(MM * NN * 2));
      const u32 eo = (u32)(((1024 * di + 16 * (49 + 2 * sub) + 4 * wv) * NN + 512 * dj) * 2 + lane * 16);
      u64 t0 = 0; u32 spins = 0;
      for (;;) {
        u32 mx = 0;
#pragma unroll
        for (int k = 0; k < 4; ++k) {
          const u32x4 v = __builtin_amdgcn_raw_buffer_load_b128(mrs, eo + k * NN * 2, 0, AUX_SYS);
          mx = max(max(mx, v.x), max(v.y, max(v.z, v.w)));
        }
        if (__all(mx != SENT)) break;
        if ((++spins & 31u) == 0) {
          const u64 now = __builtin_amdgcn_s_memrealtime();
          if (t0 == 0) t0 = now;
          if (__hip_atomic_load(tmo, __ATOMIC_RELAXED, __HIP_MEMORY_SCOPE_AGENT) == call || now - t0 > DEADLINE_TICKS) {
            __hip_atomic_store(tmo, call, __ATOMIC_RELAXED, __HIP_MEMORY_SCOPE_AGENT);
            break;
          }
        }
        __builtin_amdgcn_s_sleep(ESLEEP);
        asm volatile("" ::: "memory");
      }
      TS(12);
    }
    TS_FLUSH();
    return;
  }

  const int ri = rank >> 2, rj = rank & 3;
  const u32 cnt = __hip_atomic_load(cntp, __ATOMIC_RELAXED, __HIP_MEMORY_SCOPE_AGENT);
  const u32 call = __builtin_amdgcn_readfirstlane(cnt) + 1u;
  const int q = wv;
  const __amdgpu_buffer_rsrc_t wr = mk_rsrc(myws + WS_OFF + (size_t)q * WS_PER, (u32)WS_PER);
  const __amdgpu_buffer_rsrc_t xr = mk_rsrc(myws + XS_OFF + (size_t)q * XS_PER, (u32)XS_PER);
  const __amdgpu_buffer_rsrc_t ors = mk_rsrc(myout, (u32)(MM * NN * 2));
  const u32* wflag = (const u32*)(myws + WF_OFF + (size_t)g * FSTR) + q * 4;
  u32* sbad = (u32*)(smem + 8 * PART_BYTES);

  if ((r16 & 1) == 0 && wv < 4) {
    const u32 mo = (u32)(((1024 * ri + 16 * r16 + (tid >> 4)) * NN + 512 * rj + 64 * g + 4 * (tid & 15)) * 2);
    __builtin_amdgcn_raw_buffer_store_b64(u32x2{SENT, SENT}, ors, mo, 0, AUX_SYS);
    wait_vm<0>();
    if (lane == 0) st_flag((u32*)(myws + AF_OFF + (size_t)(r16 >> 1) * FSTR) + g * 4 + wv, call);
  }
  u32x4 wf[8][4];
  bool ok = true;
#pragma unroll
  for (int h = 0; h < 4; ++h) {
    ok = wait_flag(wflag + h, call, tmo) && ok;
    if (h & 1) TS(1 + (h >> 1));
#pragma unroll
    for (int s = 4 * (h >> 1); s < 4 * (h >> 1) + 4; ++s)
#pragma unroll
      for (int f = 2 * (h & 1); f < 2 * (h & 1) + 2; ++f)
        wf[s][f] = ld_sys(wr, (u32)(((g * 8 + s) * 4 + f) * 1024 + lane * 16));
    __builtin_amdgcn_sched_barrier(0);
  }
#pragma unroll 1
  for (int qt = 0; qt < 4; ++qt) {
    const int rf = 16 * qt + r16;
    ok = wait_flag((const u32*)(myws + XF_OFF + (size_t)(rf >> 2) * FSTR) + (rf & 3) * 8 + q, call, tmo) && ok;
    TS(3 + 3 * qt);
    u32x4 xf[8];
#pragma unroll
    for (int s = 0; s < 8; ++s) xf[s] = ld_sys(xr, (u32)((rf * 8 + s) * 1024 + lane * 16));
    __builtin_amdgcn_sched_barrier(0);
    f32x4 acc[4];
#pragma unroll
    for (int f = 0; f < 4; ++f) acc[f] = f32x4{0.f, 0.f, 0.f, 0.f};
#pragma unroll
    for (int s = 0; s < 8; ++s)
#pragma unroll
      for (int f = 0; f < 4; ++f)
        acc[f] = __builtin_amdgcn_mfma_f32_16x16x32_bf16(
            __builtin_bit_cast(bf16x8, wf[s][f]), __builtin_bit_cast(bf16x8, xf[s]), acc[f], 0, 0, 0);
    {
      char* part = smem + wv * PART_BYTES;
      const int m = lane & 15, q4 = lane >> 4;
#pragma unroll
      for (int reg = 0; reg < 4; ++reg) {
        const f32x4 v = {acc[0][reg], acc[1][reg], acc[2][reg], acc[3][reg]};
        *(f32x4*)(part + m * PROW + q4 * 64 + reg * 16) = v;
      }
      if (lane == 0) sbad[wv] = ok ? 0u : 1u;
    }
    asm volatile("s_waitcnt lgkmcnt(0)" ::: "memory");
    __builtin_amdgcn_s_barrier();
    asm volatile("" ::: "memory");
    TS(4 + 3 * qt);
    if (wv < 4) {
      const int orow = tid >> 4, oseg = tid & 15;
      f32x4 pv[8];
#pragma unroll
      for (int k = 0; k < 8; ++k) pv[k] = *(const f32x4*)(smem + k * PART_BYTES + orow * PROW + oseg * 16);
      const f32x4 s0 = ((pv[0] + pv[1]) + (pv[2] + pv[3])) + ((pv[4] + pv[5]) + (pv[6] + pv[7]));
      const u32x4 bad0 = *(const u32x4*)sbad, bad1 = *(const u32x4*)(sbad + 4);
      const bool bad = (bad0.x | bad0.y | bad0.z | bad0.w | bad1.x | bad1.y | bad1.z | bad1.w) != 0u;
      u32x2 o;
      o.x = pack_bf16(silu(s0[0]), silu(s0[1]));
      o.y = pack_bf16(silu(s0[2]), silu(s0[3]));
      if (bad || o.x == SENT) o.x = 0x7fc07fc0u;
      if (bad || o.y == SENT) o.y = 0x7fc07fc0u;
      const u32 ooff = (u32)(((1024 * ri + 16 * rf + orow) * NN + 512 * rj + 64 * g + 4 * oseg) * 2);
      __builtin_amdgcn_raw_buffer_store_b64(o, ors, ooff, 0, AUX_SYS);
      wait_vm<0>();
    }
    __builtin_amdgcn_s_barrier();
    asm volatile("" ::: "memory");
    TS(5 + 3 * qt);
    if (tid == 0) st_flag((u32*)(myws + TF_OFF) + rf * 8 + g, call);
  }
  if (tid == 0) __hip_atomic_store(cntp, call, __ATOMIC_RELAXED, __HIP_MEMORY_SCOPE_AGENT);
  TS_FLUSH();
}

extern "C" void kernel_launch(int rank, int nranks, ncclComm_t comm,
                              hipStream_t stream,
                              void* const* d_in, const int* in_sizes, int n_in,
                              void* d_out, int out_size,
                              const int* in_dtypes, int out_dtype,
                              void* const* d_out_all,
                              void* const* d_ws_all, size_t ws_bytes) {
  (void)comm; (void)in_sizes; (void)n_in; (void)d_out; (void)out_size; (void)in_dtypes; (void)out_dtype;
  if (nranks != NR || (unsigned)rank >= (unsigned)NR || ws_bytes < WS_NEEDED) {
    fprintf(stderr, "unsupported configuration\n");
    return;
  }
  Params p;
  p.x = (const u16*)d_in[0];
  p.w = (const u16*)d_in[1];
  for (int i = 0; i < NR; ++i) { p.out[i] = (char*)d_out_all[i]; p.ws[i] = (char*)d_ws_all[i]; }
  p.rank = rank;
  p.myws = (char*)d_ws_all[rank];
  p.myout = (char*)d_out_all[rank];
  p.pad_[0] = p.pad_[1] = p.pad_[2] = 0;
  gemm_ar_silu<<<GRID, 512, 0, stream>>>(p);
  HIP_CHECK(hipGetLastError());
}
```

```cpp
#include <hip/hip_runtime.h>
#include <rccl/rccl.h>
#include <cstdio>
#include <cstdint>

#define HIP_CHECK(expr) do { hipError_t _e = (expr); if (_e != hipSuccess) { \
    fprintf(stderr, "HIP error %s at %s:%d\n", hipGetErrorString(_e), __FILE__, __LINE__); \
    return; } } while (0)

typedef unsigned int u32;
typedef unsigned long long u64;
typedef unsigned short u16;
typedef __attribute__((ext_vector_type(4))) unsigned int u32x4;
typedef __attribute__((ext_vector_type(2))) unsigned int u32x2;
typedef __attribute__((ext_vector_type(8))) short bf16x8;
typedef __attribute__((ext_vector_type(4))) float f32x4;

#define PSLEEP 4
#define DEPTH 2
static_assert(DEPTH >= 2 && DEPTH <= 4, "");
#define ESLEEP 1
#define DSLEEP 4
constexpr int NR = 8;
constexpr int MM = 2048, NN = 2048, KSH = 256;
constexpr int PPD = 8;
constexpr int NPUSH = PPD * NR;
constexpr int NT = 6;
constexpr int NGEMM = 128;
constexpr int GRID = NPUSH + NGEMM;
static_assert(GRID <= 256, "all workgroups must be co-resident");

constexpr size_t XS_PER = 512u << 10;
constexpr size_t WS_PER = 256u << 10;
constexpr size_t XS_OFF = 0;
constexpr size_t WS_OFF = NR * XS_PER;
constexpr size_t SLOT_BYTES = NR * (XS_PER + WS_PER);
constexpr size_t FLAG_OFF = SLOT_BYTES;
constexpr size_t FSTR = 128;
constexpr size_t WF_OFF = FLAG_OFF;
constexpr size_t XF_OFF = WF_OFF + 8 * FSTR;
constexpr size_t TF_OFF = XF_OFF + 16 * FSTR;
constexpr size_t DF_OFF = TF_OFF + 32 * FSTR;
constexpr size_t AF_OFF = DF_OFF + 32 * FSTR;
constexpr size_t CNT_OFF = AF_OFF + 8 * FSTR;
constexpr size_t TMO_OFF = CNT_OFF + (size_t)GRID * FSTR;
constexpr size_t WS_NEEDED = TMO_OFF + FSTR;

constexpr u64 DEADLINE_TICKS = 40000000ull;

#define TS(i) do {} while (0)
#define TS_FLUSH() do {} while (0)

struct Params {
  const u16* x;
  const u16* w;
  char* myws;
  char* myout;
  int rank;
  u32 dmap;
  int pad_[2];
  char* ws[NR];
  char* out[NR];
};

constexpr int AUX_SYS = 17;
#define AUX_LD 17

__device__ __forceinline__ __amdgpu_buffer_rsrc_t mk_rsrc(const void* p, u32 bytes) {
  return __builtin_amdgcn_make_buffer_rsrc((void*)p, 0, bytes, 0x00020000);
}
__device__ __forceinline__ void st_sys(u32x4 v, __amdgpu_buffer_rsrc_t r, u32 off) {
  __builtin_amdgcn_raw_buffer_store_b128(v, r, off, 0, AUX_SYS);
}
__device__ __forceinline__ u32x4 ld_sys(__amdgpu_buffer_rsrc_t r, u32 off) {
  return __builtin_amdgcn_raw_buffer_load_b128(r, off, 0, AUX_LD);
}

typedef __attribute__((address_space(1))) u32 gu32;
__device__ __forceinline__ void st_flag(u32* f, u32 v) {
  __hip_atomic_store((gu32*)(size_t)f, v, __ATOMIC_RELAXED, __HIP_MEMORY_SCOPE_SYSTEM);
}

__device__ __forceinline__ bool wait_ge(const u32* f, u32 thr, u32 call, u32* tmo) {
  u64 t0 = 0;
  u32 spins = 0;
  for (;;) {
    const u32 v = __hip_atomic_load(f, __ATOMIC_RELAXED, __HIP_MEMORY_SCOPE_SYSTEM);
    if (__all(v >= thr)) break;
    if ((++spins & 31u) == 0) {
      const u64 now = __builtin_amdgcn_s_memrealtime();
      if (t0 == 0) t0 = now;
      if (__hip_atomic_load(tmo, __ATOMIC_RELAXED, __HIP_MEMORY_SCOPE_AGENT) == call ||
          now - t0 > DEADLINE_TICKS) {
        __hip_atomic_store(tmo, call, __ATOMIC_RELAXED, __HIP_MEMORY_SCOPE_AGENT);
        return false;
      }
    }
    __builtin_amdgcn_s_sleep(PSLEEP);
  }
  asm volatile("" ::: "memory");
  return true;
}

__device__ __forceinline__ bool wait_flag(const u32* f, u32 call, u32* tmo) { return wait_ge(f, call, call, tmo); }

template <int N> __device__ __forceinline__ void wait_vm() {
  asm volatile("s_waitcnt vmcnt(%0)" :: "i"(N) : "memory");
}

__device__ __forceinline__ u32 pack_bf16(float a, float b) {
  u32 r;
  asm("v_cvt_pk_bf16_f32 %0, %1, %2" : "=v"(r) : "v"(a), "v"(b));
  return r;
}
__device__ __forceinline__ float silu(float y) {
  const float yc = fminf(fmaxf(y, -60.0f), 60.0f);
  return y / (1.0f + __expf(-yc));
}

constexpr int PROW = 272;
constexpr int PART_BYTES = 16 * PROW;
constexpr int LDS_BYTES = 8 * PART_BYTES + 64;

constexpr bool round_is_w(int n) { return n < 2; }

constexpr int pace_cnt(int j) {
  int x = DEPTH - 1;
  for (int n = 0; n + 2 < NT; ++n)
    if (j - DEPTH <= 4 * n + 3 && 4 * n + 3 <= j - 1) x += 4;
  return x;
}
template <int I> struct IC { static constexpr int value = I; };
template <int N, int I = 0, class F> __device__ __forceinline__ void static_for(F&& f) {
  if constexpr (I < N) { f(IC<I>{}); static_for<N, I + 1>(f); }
}

template <int D>
__device__ __forceinline__ void store4(int sel, u32x4 (&v)[4], u32 vo, u32x4 rs, u32 s0, u32 s1, u32 s2, u32 s3) {
  constexpr int E = D - 1 < 4 ? D - 1 : 4;
  asm volatile("s_cmp_lt_u32 %10, 2\n\t"
               "s_cbranch_scc1 .Lnp_%=\n\t"
               "s_cmp_eq_u32 %10, 3\n\t"
               "s_cbranch_scc1 .Lw3_%=\n\t"
               "s_waitcnt vmcnt(4)\n"
               ".Lw3_%=:\n\t"
               "s_waitcnt vmcnt(%13)\n\t"
               "buffer_store_dwordx4 %0, %4, %5, %6 offen sc0 sc1\n\t"
               "s_waitcnt vmcnt(%14)\n\t"
               "buffer_store_dwordx4 %1, %4, %5, %7 offen sc0 sc1\n\t"
               "s_waitcnt vmcnt(%15)\n\t"
               "buffer_store_dwordx4 %2, %4, %5, %8 offen sc0 sc1\n\t"
               "s_waitcnt vmcnt(%16)\n\t"
               "s_branch .Ltail_%=\n"
               ".Lnp_%=:\n\t"
               "s_cmp_eq_u32 %10, 1\n\t"
               "s_cbranch_scc1 .Lw1_%=\n\t"
               "s_waitcnt vmcnt(0)\n"
               ".Lw1_%=:\n\t"
               "s_waitcnt vmcnt(%12)\n\t"
               "buffer_store_dwordx4 %0, %4, %5, %6 offen sc0 sc1\n\t"
               "s_waitcnt vmcnt(%11)\n\t"
               "buffer_store_dwordx4 %1, %4, %5, %7 offen sc0 sc1\n\t"
               "s_waitcnt vmcnt(%11)\n\t"
               "buffer_store_dwordx4 %2, %4, %5, %8 offen sc0 sc1\n\t"
               "s_waitcnt vmcnt(%11)\n"
               ".Ltail_%=:\n\t"
               "buffer_store_dwordx4 %3, %4, %5, %9 offen sc0 sc1\n\t"
               "s_nop 1"
               :: "v"(v[0]), "v"(v[1]), "v"(v[2]), "v"(v[3]),
                  "v"(vo), "s"(rs), "s"(s0), "s"(s1), "s"(s2), "s"(s3),
                  "s"(sel), "i"(D - 1), "i"(E), "i"(E + 4),
                  "i"(1 <= D - 1 ? D + 3 : D - 1), "i"(2 <= D - 1 ? D + 3 : D - 1), "i"(3 <= D - 1 ? D + 3 : D - 1)
               : "scc", "memory");
}

constexpr u32 SENT = 0xffffffffu;
template <int N>
__device__ __forceinline__ bool valid4(u32x4 (&v)[4]) {
  u64 m; u32 t;
  asm volatile("s_waitcnt vmcnt(%18)\n\t"
               "v_max3_u32 %1, %2, %3, %4\n\t"
               "v_max3_u32 %1, %1, %5, %6\n\t"
               "v_max3_u32 %1, %1, %7, %8\n\t"
               "v_max3_u32 %1, %1, %9, %10\n\t"
               "v_max3_u32 %1, %1, %11, %12\n\t"
               "v_max3_u32 %1, %1, %13, %14\n\t"
               "v_max3_u32 %1, %1, %15, %16\n\t"
               "v_max_u32 %1, %1, %17\n\t"
               "v_cmp_eq_u32_e64 %0, -1, %1\n\t"
               "s_nop 4"
               : "=s"(m), "=&v"(t)
               : "v"(v[0].x), "v"(v[0].y), "v"(v[0].z), "v"(v[0].w), "v"(v[1].x), "v"(v[1].y), "v"(v[1].z), "v"(v[1].w),
                 "v"(v[2].x), "v"(v[2].y), "v"(v[2].z), "v"(v[2].w), "v"(v[3].x), "v"(v[3].y), "v"(v[3].z), "v"(v[3].w),
                 "i"(N)
               : "memory");
  return m == 0ull;
}

__global__ void __launch_bounds__(512, 2) gemm_ar_silu(Params p) {
  __shared__ __attribute__((aligned(16))) char smem[LDS_BYTES];
  const int tid = threadIdx.x, lane = tid & 63;
  const int wv = __builtin_amdgcn_readfirstlane(tid >> 6);
  const int bid = blockIdx.x;
  const bool is_engine = bid < NPUSH;
  const int xs = bid & 7;
  const int sub = bid >> 3;
#define GMAP 2
  const int gs = (bid - NPUSH) >> 3;
  const int g = ((xs & ((8 >> GMAP) - 1)) << GMAP) + (gs & ((1 << GMAP) - 1));
  const int r16 = (xs >> (3 - GMAP)) * (16 >> GMAP) + (gs >> GMAP);
  u32 dmap = p.dmap;
  int rank = p.rank;
  u64 a0 = (u64)p.myws, a1 = (u64)p.myout, a2 = (u64)p.x, a3 = (u64)p.w, a4 = (u64)p.ws[xs], a5 = (u64)p.out[xs];
  asm volatile("" : "+s"(rank), "+s"(dmap), "+s"(a0), "+s"(a1), "+s"(a2), "+s"(a3), "+s"(a4), "+s"(a5));
  const int dst = (dmap >> (4 * xs)) & 7;
  typedef __attribute__((address_space(1))) char gchar;
  char* const myws = (char*)(gchar*)a0;
  char* const myout = (char*)(gchar*)a1;
  const u16* const px = (const u16*)(char*)(gchar*)a2;
  const u16* const pw = (const u16*)(char*)(gchar*)a3;
  char* const dws = (char*)(gchar*)a4;
  char* const dout = (char*)(gchar*)a5;
  u32* cntp = (u32*)(myws + CNT_OFF + (size_t)bid * FSTR);
  u32* tmo = (u32*)(myws + TMO_OFF);
  TS(0);

  if (__builtin_expect(is_engine, 1)) {
    if (wv > 5) return;
    const int di = dst >> 2, dj = dst & 3;
    u32* lcnt = (u32*)smem;
    if (wv >= 4) {
      const u32 cnt = __hip_atomic_load(cntp, __ATOMIC_RELAXED, __HIP_MEMORY_SCOPE_AGENT);
      __builtin_amdgcn_s_setprio(3);
      const u32 call = __builtin_amdgcn_readfirstlane(cnt) + 1u;
      if (wv == 4 && lane < 17) lcnt[lane] = lane == 16 ? call : 0u;
      asm volatile("s_waitcnt lgkmcnt(0)" :: "s"(call) : "memory");
      __builtin_amdgcn_s_barrier();
      asm volatile("" ::: "memory");
      if (wv >= 5) {
        if (dst == rank) return;
        {
          const __amdgpu_buffer_rsrc_t mrs = mk_rsrc(myout, (u32)(MM * NN * 2));
          const u32 eo = (u32)(((1024 * di + 16 * (49 + 2 * sub)) * NN + 512 * dj) * 2 + lane * 16);
#pragma unroll
          for (int k = 0; k < 16; ++k) st_sys(u32x4{SENT, SENT, SENT, SENT}, mrs, eo + k * NN * 2);
          wait_vm<0>();
          if (lane == 0) __hip_atomic_store(lcnt + 14, 1u, __ATOMIC_RELAXED, __HIP_MEMORY_SCOPE_WORKGROUP);
        }
        if (wait_flag((const u32*)(myws + AF_OFF + (size_t)sub * FSTR) + (lane & 31), call, tmo) && lane == 0)
          __hip_atomic_fetch_or(lcnt + 12, 0x100u, __ATOMIC_RELAXED, __HIP_MEMORY_SCOPE_WORKGROUP);
        while (__hip_atomic_load(lcnt + 4, __ATOMIC_RELAXED, __HIP_MEMORY_SCOPE_WORKGROUP) < 4u)
          __builtin_amdgcn_s_sleep(4);
        const u32* tf = (const u32*)(myws + TF_OFF) + (16 * (lane >> 4) + 2 * sub + ((lane >> 3) & 1)) * 8 + (lane & 7);
        u64 t0 = 0; u32 spins = 0, prev = 0;
        for (;;) {
          const u32 v = __hip_atomic_load(tf, __ATOMIC_RELAXED, __HIP_MEMORY_SCOPE_SYSTEM);
          const u64 bm = __ballot(v >= call);
          u32 rm = 0;
#pragma unroll
          for (int m = 0; m < 8; ++m) rm |= (((bm >> (8 * m)) & 0xffull) == 0xffull) ? (1u << m) : 0u;
          if (rm != prev) {
            if (lane == 0) __hip_atomic_fetch_or(lcnt + 12, rm, __ATOMIC_RELAXED, __HIP_MEMORY_SCOPE_WORKGROUP);
            prev = rm;
          }
          if (rm == 0xffu) break;
          if ((++spins & 31u) == 0) {
            const u64 now = __builtin_amdgcn_s_memrealtime();
            if (t0 == 0) t0 = now;
            if (__hip_atomic_load(tmo, __ATOMIC_RELAXED, __HIP_MEMORY_SCOPE_AGENT) == call || now - t0 > DEADLINE_TICKS) {
              __hip_atomic_store(tmo, call, __ATOMIC_RELAXED, __HIP_MEMORY_SCOPE_AGENT);
              if (lane == 0) {
                __hip_atomic_store(lcnt + 13, 1u, __ATOMIC_RELAXED, __HIP_MEMORY_SCOPE_WORKGROUP);
                asm volatile("s_waitcnt lgkmcnt(0)" ::: "memory");
                __hip_atomic_fetch_or(lcnt + 12, 0xffu, __ATOMIC_RELAXED, __HIP_MEMORY_SCOPE_WORKGROUP);
              }
              break;
            }
          }
          __builtin_amdgcn_s_sleep(PSLEEP);
        }
        TS(10);
        TS_FLUSH();
        return;
      }
#pragma unroll
      for (int h = 0; h < 2 * NT; ++h) {
        u32* fp;
        if (h < 4) {
          fp = (u32*)(dws + WF_OFF + (size_t)sub * FSTR) + (rank * 4 + h);
        } else {
          const int rfi = 2 * (8 * ((h - 4) >> 1) + sub) + (h & 1);
          fp = (u32*)(dws + XF_OFF + (size_t)(rfi >> 2) * FSTR) + ((rfi & 3) * 8 + rank);
        }
        while (__hip_atomic_load(lcnt + h, __ATOMIC_RELAXED, __HIP_MEMORY_SCOPE_WORKGROUP) < 4u)
          __builtin_amdgcn_s_sleep(1);
        if (h == 3 && dst != rank)
          while (__hip_atomic_load(lcnt + 14, __ATOMIC_RELAXED, __HIP_MEMORY_SCOPE_WORKGROUP) == 0u)
            __builtin_amdgcn_s_sleep(1);
        asm volatile("" ::: "memory");
        if (lane == 0) st_flag(fp, call);
        if (h & 1) TS(2 + (h >> 1));
      }
      if (dst != rank) {
        while (__hip_atomic_load(lcnt + 15, __ATOMIC_RELAXED, __HIP_MEMORY_SCOPE_WORKGROUP) < 4u)
          __builtin_amdgcn_s_sleep(1);
        asm volatile("" ::: "memory");
        TS(14);
        if (lane == 0) st_flag((u32*)(dws + DF_OFF + (size_t)sub * FSTR) + rank, call);
        const bool ok2 = wait_flag((const u32*)(myws + DF_OFF + (size_t)sub * FSTR) + dst, call, tmo);
        if (!ok2 && lane == 0) {
          ((u16*)myout)[(size_t)(1024 * (rank >> 2)) * NN + 512 * (rank & 3)] = 0x7fc0;
        }
      }
      if (lane == 0) __hip_atomic_store(cntp, call, __ATOMIC_RELAXED, __HIP_MEMORY_SCOPE_AGENT);
      TS(15);
      TS_FLUSH();
      return;
    }
    const int c = lane & 15, kg = lane >> 4;
    u32 r[NT][16];
    auto load_task = [&](int n) {
      if (round_is_w(n)) {
        const int kh = n, g = sub, ks = 4 * kh + wv;
        const u16* src = pw + (size_t)(32 * ks + 8 * kg) * NN + 512 * dj + 64 * g + 4 * c;
#pragma unroll
        for (int jj = 0; jj < 8; ++jj) {
          const u32x2 v = *(const u32x2*)(src + (size_t)jj * NN);
          r[n][2 * jj] = v.x; r[n][2 * jj + 1] = v.y;
        }
      } else {
        const int rfi = 2 * (8 * (n - 2) + sub);
        const u16* src = px + (size_t)(1024 * di + 16 * rfi + c) * KSH + 64 * wv + 8 * kg;
#pragma unroll
        for (int k = 0; k < 4; ++k) {
          const u32x4 v = *(const u32x4*)(src + (k >> 1) * 16 * KSH + 32 * (k & 1));
          r[n][4 * k] = v.x; r[n][4 * k + 1] = v.y; r[n][4 * k + 2] = v.z; r[n][4 * k + 3] = v.w;
        }
      }
    };
    auto transpose = [&](int n) {
      u32 f[16];
#pragma unroll
      for (int d = 0; d < 4; ++d) {
        const u32 a0 = r[n][4 * d], a1 = r[n][4 * d + 1], b0 = r[n][4 * d + 2], b1 = r[n][4 * d + 3];
        f[0 + d] = __builtin_amdgcn_perm(b0, a0, 0x05040100u);
        f[4 + d] = __builtin_amdgcn_perm(b0, a0, 0x07060302u);
        f[8 + d] = __builtin_amdgcn_perm(b1, a1, 0x05040100u);
        f[12 + d] = __builtin_amdgcn_perm(b1, a1, 0x07060302u);
      }
#pragma unroll
      for (int e = 0; e < 16; ++e) r[n][e] = f[e];
    };
    const __amdgpu_buffer_rsrc_t wrs = mk_rsrc(dws + WS_OFF + (size_t)rank * WS_PER, (u32)WS_PER);
    const __amdgpu_buffer_rsrc_t xrs = mk_rsrc(dws + XS_OFF + (size_t)rank * XS_PER, (u32)XS_PER);
    auto store_k = [&](int n, int k) {
      const u32x4 v = {r[n][4 * k], r[n][4 * k + 1], r[n][4 * k + 2], r[n][4 * k + 3]};
      if (round_is_w(n)) {
        const int kh = n, g = sub, ks = 4 * kh + wv;
        st_sys(v, wrs, (u32)(((g * 8 + ks) * 4 + k) * 1024 + lane * 16));
      } else {
        const int rfi = 2 * (8 * (n - 2) + sub) + (k >> 1);
        st_sys(v, xrs, (u32)((rfi * 8 + 2 * wv + (k & 1)) * 1024 + lane * 16));
      }
    };
    auto signal = [&](int n, int cnt = 1) {
      if (lane < cnt) __hip_atomic_fetch_add(lcnt + n + lane, 1u, __ATOMIC_RELAXED, __HIP_MEMORY_SCOPE_WORKGROUP);
    };
    constexpr int HS = (4 * (NT - 1) - DEPTH) / 2;
    load_task(0);
    __builtin_amdgcn_sched_barrier(0);
    load_task(1);
    __builtin_amdgcn_sched_barrier(0);
    TS(1);
    transpose(0);
    static_for<4 * (NT - 1)>([&](auto J) {
      constexpr int j = decltype(J)::value;
      if (j == DEPTH) {
        transpose(1);
        __builtin_amdgcn_s_barrier();
        asm volatile("" ::: "memory");
      }
      if (j >= DEPTH) {
        wait_vm<pace_cnt(j)>();
        if (((j - DEPTH) & 1) == 1) signal((j - DEPTH) >> 1);
        if (((j - DEPTH) & 3) == 3) TS(2 + ((j - DEPTH) >> 2));
      }
      store_k(j >> 2, j & 3);
      __builtin_amdgcn_sched_barrier(0);
      if ((j & 3) == 3 && (j >> 2) + 2 < NT) { load_task((j >> 2) + 2); __builtin_amdgcn_sched_barrier(0); }
    });
    const int ri = rank >> 2, rj = rank & 3;
    const u64 a6 = a4 + XS_OFF + (u64)rank * XS_PER;
    const u32x4 xrs4 = {(u32)a6, (u32)(a6 >> 32) & 0xffffu, (u32)XS_PER, 0x00020000u};
    const u32x4 srs = {(u32)a1, (u32)(a1 >> 32) & 0xffffu, (u32)(MM * NN * 2), 0x00020000u};
    const u32x4 drs = {(u32)a5, (u32)(a5 >> 32) & 0xffffu, (u32)(MM * NN * 2), 0x00020000u};
    const u32 off0 = (u32)(((1024 * ri + 32 * sub + 4 * wv) * NN + 512 * rj) * 2 + lane * 16);
    auto moff = [&](int m) -> u32 { return (u32)(16 * (m >> 1) + (m & 1)) * (u32)(16 * NN * 2); };
    auto ready = [&](int m) -> bool {
      return (__hip_atomic_load(lcnt + 12, __ATOMIC_RELAXED, __HIP_MEMORY_SCOPE_WORKGROUP) >> m) & 1u;
    };
    auto spin = [&](int m) { while (!ready(m)) __builtin_amdgcn_s_sleep(1); asm volatile("" ::: "memory"); };
    auto ld4 = [&](int m, u32x4 (&v)[4]) {
      const u32 so = moff(m);
      asm volatile("buffer_load_dwordx4 %0, %4, %5, %6 offen sc0 sc1\n\t"
                   "buffer_load_dwordx4 %1, %4, %5, %7 offen sc0 sc1\n\t"
                   "buffer_load_dwordx4 %2, %4, %5, %8 offen sc0 sc1\n\t"
                   "buffer_load_dwordx4 %3, %4, %5, %9 offen sc0 sc1"
                   : "=&v"(v[0]), "=&v"(v[1]), "=&v"(v[2]), "=&v"(v[3])
                   : "v"(off0), "s"(srs), "s"(so), "s"(so + NN * 2), "s"(so + 2 * NN * 2), "s"(so + 3 * NN * 2)
                   : "memory");
    };
    u32x4 A[4], B[4];
    bool early;
    {
      u32x4 X[4];
#pragma unroll
      for (int k = 0; k < 4; ++k) X[k] = u32x4{r[NT - 1][4 * k], r[NT - 1][4 * k + 1], r[NT - 1][4 * k + 2], r[NT - 1][4 * k + 3]};
      asm volatile("" : "+v"(X[0]), "+v"(X[1]), "+v"(X[2]), "+v"(X[3]) :: "memory");
      early = dst != rank && ready(8);
      if (early) { asm volatile("" ::: "memory"); ld4(0, A); }
      const u32 xo = (u32)((2 * (8 * (NT - 3) + sub) * 8 + 2 * wv) * 1024 + lane * 16);
      store4<DEPTH>(__builtin_amdgcn_readfirstlane(early ? 3 : 1), X, xo, xrs4, 0u, 1024u, 8192u, 9216u);
      signal(HS, 2 * NT - 2 - HS);
      TS(2 + NT - 2);
      if (early) early = valid4<4>(A);
      if (dst == rank) {
        wait_vm<0>();
        signal(2 * NT - 2, 2);
        TS(2 + NT - 1);
      }
    }
    if (dst != rank) {
      const bool late5 = early;
      if (!early) {
        bool first = true;
        for (;;) {
          u32 st = __hip_atomic_load(lcnt + 12, __ATOMIC_RELAXED, __HIP_MEMORY_SCOPE_WORKGROUP);
          if (st & 0x101u) {
            asm volatile("" ::: "memory");
            ld4(0, A);
          }
          const bool okv = valid4<0>(A);
          if (first) { signal(2 * NT - 2, 2); TS(2 + NT - 1); first = false; }
          if ((st & 1u) || ((st & 0x100u) && okv)) break;
          __builtin_amdgcn_s_sleep(DSLEEP);
        }
      }
      TS(8);
      auto step = [&](auto DD, int m, u32x4 (&cur)[4], u32x4 (&nxt)[4]) {
        const bool more = m + 1 < 8;
        const bool pre = more && ready(m + 1);
        if (pre) { asm volatile("" ::: "memory"); ld4(m + 1, nxt); }
        const u32 so = moff(m);
        store4<decltype(DD)::value>(__builtin_amdgcn_readfirstlane((early ? 1 : 0) + (pre ? 2 : 0)), cur, off0, drs,
                                    so, so + NN * 2, so + 2 * NN * 2, so + 3 * NN * 2);
        if (m == 0 && late5) { signal(2 * NT - 2, 2); TS(2 + NT - 1); }
        if (more && !pre) { spin(m + 1); ld4(m + 1, nxt); }
        early = pre;
      };
#pragma unroll 1
      for (int m = 0; m < 6; m += 2) { step(IC<DEPTH>{}, m, A, B); step(IC<DEPTH>{}, m + 1, B, A); }
      step(IC<DEPTH>{}, 6, A, B);
      {
        const bool bad = __hip_atomic_load(lcnt + 13, __ATOMIC_RELAXED, __HIP_MEMORY_SCOPE_WORKGROUP) != 0u;
        const u32 so = moff(7);
        if (early) wait_vm<DEPTH - 1>(); else wait_vm<0>();
#pragma unroll
        for (int k = 0; k < 4; ++k) {
          if (k > 0) wait_vm<DEPTH - 1>();
          if (k == DEPTH - 1 && !bad) { signal(15); TS(13); }
          asm volatile("buffer_store_dwordx4 %0, %1, %2, %3 offen sc0 sc1\n\ts_nop 1"
                       :: "v"(B[k]), "v"(off0), "s"(drs), "s"(so + k * NN * 2) : "memory");
        }
        if (bad) {
          wait_vm<0>();
          const __amdgpu_buffer_rsrc_t prs = mk_rsrc(dout, (u32)(MM * NN * 2));
          for (int m = 0; m < 8; ++m)
            for (int k = 0; k < 4; ++k)
              st_sys(u32x4{0x7fc07fc0u, 0x7fc07fc0u, 0x7fc07fc0u, 0x7fc07fc0u}, prs, off0 + moff(m) + k * NN * 2);
          wait_vm<0>();
          signal(15);
        }
      }
      const u32 call = __builtin_amdgcn_readfirstlane(lcnt[16]);
      const __amdgpu_buffer_rsrc_t mrs = mk_rsrc(myout, (u32)(MM * NN * 2));
      const u32 eo = (u32)(((1024 * di + 16 * (49 + 2 * sub) + 4 * wv) * NN + 512 * dj) * 2 + lane * 16);
      u64 t0 = 0; u32 spins = 0;
      for (;;) {
        u32 mx = 0;
#pragma unroll
        for (int k = 0; k < 4; ++k) {
          const u32x4 v = __builtin_amdgcn_raw_buffer_load_b128(mrs, eo + k * NN * 2, 0, AUX_SYS);
          mx = max(max(mx, v.x), max(v.y, max(v.z, v.w)));
        }
        if (__all(mx != SENT)) break;
        if ((++spins & 31u) == 0) {
          const u64 now = __builtin_amdgcn_s_memrealtime();
          if (t0 == 0) t0 = now;
          if (__hip_atomic_load(tmo, __ATOMIC_RELAXED, __HIP_MEMORY_SCOPE_AGENT) == call || now - t0 > DEADLINE_TICKS) {
            __hip_atomic_store(tmo, call, __ATOMIC_RELAXED, __HIP_MEMORY_SCOPE_AGENT);
            break;
          }
        }
        __builtin_amdgcn_s_sleep(ESLEEP);
        asm volatile("" ::: "memory");
      }
      TS(12);
    }
    TS_FLUSH();
    return;
  }

  const int ri = rank >> 2, rj = rank & 3;
  const u32 cnt = __hip_atomic_load(cntp, __ATOMIC_RELAXED, __HIP_MEMORY_SCOPE_AGENT);
  const u32 call = __builtin_amdgcn_readfirstlane(cnt) + 1u;
  const int q = wv;
  const __amdgpu_buffer_rsrc_t wr = mk_rsrc(myws + WS_OFF + (size_t)q * WS_PER, (u32)WS_PER);
  const __amdgpu_buffer_rsrc_t xr = mk_rsrc(myws + XS_OFF + (size_t)q * XS_PER, (u32)XS_PER);
  const __amdgpu_buffer_rsrc_t ors = mk_rsrc(myout, (u32)(MM * NN * 2));
  const u32* wflag = (const u32*)(myws + WF_OFF + (size_t)g * FSTR) + q * 4;
  u32* sbad = (u32*)(smem + 8 * PART_BYTES);

  if ((r16 & 1) == 0 && wv < 4) {
    const u32 mo = (u32)(((1024 * ri + 16 * r16 + (tid >> 4)) * NN + 512 * rj + 64 * g + 4 * (tid & 15)) * 2);
    __builtin_amdgcn_raw_buffer_store_b64(u32x2{SENT, SENT}, ors, mo, 0, AUX_SYS);
    wait_vm<0>();
    if (lane == 0) st_flag((u32*)(myws + AF_OFF + (size_t)(r16 >> 1) * FSTR) + g * 4 + wv, call);
  }
  u32x4 wf[8][4];
  bool ok = true;
#pragma unroll
  for (int h = 0; h < 4; ++h) {
    ok = wait_flag(wflag + h, call, tmo) && ok;
    if (h & 1) TS(1 + (h >> 1));
#pragma unroll
    for (int s = 4 * (h >> 1); s < 4 * (h >> 1) + 4; ++s)
#pragma unroll
      for (int f = 2 * (h & 1); f < 2 * (h & 1) + 2; ++f)
        wf[s][f] = ld_sys(wr, (u32)(((g * 8 + s) * 4 + f) * 1024 + lane * 16));
    __builtin_amdgcn_sched_barrier(0);
  }
#pragma unroll 1
  for (int qt = 0; qt < 4; ++qt) {
    const int rf = 16 * qt + r16;
    ok = wait_flag((const u32*)(myws + XF_OFF + (size_t)(rf >> 2) * FSTR) + (rf & 3) * 8 + q, call, tmo) && ok;
    TS(3 + 3 * qt);
    u32x4 xf[8];
#pragma unroll
    for (int s = 0; s < 8; ++s) xf[s] = ld_sys(xr, (u32)((rf * 8 + s) * 1024 + lane * 16));
    __builtin_amdgcn_sched_barrier(0);
    f32x4 acc[4];
#pragma unroll
    for (int f = 0; f < 4; ++f) acc[f] = f32x4{0.f, 0.f, 0.f, 0.f};
#pragma unroll
    for (int s = 0; s < 8; ++s)
#pragma unroll
      for (int f = 0; f < 4; ++f)
        acc[f] = __builtin_amdgcn_mfma_f32_16x16x32_bf16(
            __builtin_bit_cast(bf16x8, wf[s][f]), __builtin_bit_cast(bf16x8, xf[s]), acc[f], 0, 0, 0);
    {
      char* part = smem + wv * PART_BYTES;
      const int m = lane & 15, q4 = lane >> 4;
#pragma unroll
      for (int reg = 0; reg < 4; ++reg) {
        const f32x4 v = {acc[0][reg], acc[1][reg], acc[2][reg], acc[3][reg]};
        *(f32x4*)(part + m * PROW + q4 * 64 + reg * 16) = v;
      }
      if (lane == 0) sbad[wv] = ok ? 0u : 1u;
    }
    asm volatile("s_waitcnt lgkmcnt(0)" ::: "memory");
    __builtin_amdgcn_s_barrier();
    asm volatile("" ::: "memory");
    TS(4 + 3 * qt);
    if (wv < 4) {
      const int orow = tid >> 4, oseg = tid & 15;
      f32x4 pv[8];
#pragma unroll
      for (int k = 0; k < 8; ++k) pv[k] = *(const f32x4*)(smem + k * PART_BYTES + orow * PROW + oseg * 16);
      const f32x4 s0 = ((pv[0] + pv[1]) + (pv[2] + pv[3])) + ((pv[4] + pv[5]) + (pv[6] + pv[7]));
      const u32x4 bad0 = *(const u32x4*)sbad, bad1 = *(const u32x4*)(sbad + 4);
      const bool bad = (bad0.x | bad0.y | bad0.z | bad0.w | bad1.x | bad1.y | bad1.z | bad1.w) != 0u;
      u32x2 o;
      o.x = pack_bf16(silu(s0[0]), silu(s0[1]));
      o.y = pack_bf16(silu(s0[2]), silu(s0[3]));
      if (bad || o.x == SENT) o.x = 0x7fc07fc0u;
      if (bad || o.y == SENT) o.y = 0x7fc07fc0u;
      const u32 ooff = (u32)(((1024 * ri + 16 * rf + orow) * NN + 512 * rj + 64 * g + 4 * oseg) * 2);
      __builtin_amdgcn_raw_buffer_store_b64(o, ors, ooff, 0, AUX_SYS);
      wait_vm<0>();
    }
    __builtin_amdgcn_s_barrier();
    asm volatile("" ::: "memory");
    TS(5 + 3 * qt);
    if (tid == 0) st_flag((u32*)(myws + TF_OFF) + rf * 8 + g, call);
  }
  if (tid == 0) __hip_atomic_store(cntp, call, __ATOMIC_RELAXED, __HIP_MEMORY_SCOPE_AGENT);
  TS_FLUSH();
}

extern "C" void kernel_launch(int rank, int nranks, ncclComm_t comm,
                              hipStream_t stream,
                              void* const* d_in, const int* in_sizes, int n_in,
                              void* d_out, int out_size,
                              const int* in_dtypes, int out_dtype,
                              void* const* d_out_all,
                              void* const* d_ws_all, size_t ws_bytes) {
  (void)comm; (void)in_sizes; (void)n_in; (void)d_out; (void)out_size; (void)in_dtypes; (void)out_dtype;
  if (nranks != NR || (unsigned)rank >= (unsigned)NR || ws_bytes < WS_NEEDED) {
    fprintf(stderr, "unsupported configuration\n");
    return;
  }
  Params p;
  p.x = (const u16*)d_in[0];
  p.w = (const u16*)d_in[1];
  constexpr unsigned QUAD1 = 0xb2u;
  const unsigned myq = (QUAD1 >> rank) & 1u;
  int lo = 0, hi = 4;
  p.dmap = 0;
  for (int d = 0; d < NR; ++d) {
    const int x = (((QUAD1 >> d) & 1u) == myq) ? lo++ : hi++;
    p.dmap |= (u32)d << (4 * x);
    p.out[x] = (char*)d_out_all[d];
    p.ws[x] = (char*)d_ws_all[d];
  }
  p.rank = rank;
  p.myws = (char*)d_ws_all[rank];
  p.myout = (char*)d_out_all[rank];
  p.pad_[0] = p.pad_[1] = 0;
  gemm_ar_silu<<<GRID, 512, 0, stream>>>(p);
  HIP_CHECK(hipGetLastError());
}
```

```cpp
#include <hip/hip_runtime.h>
#include <rccl/rccl.h>
#include <cstdio>
#include <cstdint>

#define HIP_CHECK(expr) do { hipError_t _e = (expr); if (_e != hipSuccess) { \
    fprintf(stderr, "HIP error %s at %s:%d\n", hipGetErrorString(_e), __FILE__, __LINE__); \
    return; } } while (0)

typedef unsigned int u32;
typedef unsigned long long u64;
typedef unsigned short u16;
typedef __attribute__((ext_vector_type(4))) unsigned int u32x4;
typedef __attribute__((ext_vector_type(2))) unsigned int u32x2;
typedef __attribute__((ext_vector_type(8))) short bf16x8;
typedef __attribute__((ext_vector_type(4))) float f32x4;
typedef __attribute__((ext_vector_type(2))) float f32x2;

#define PSLEEP 4
#define DEPTH 2
static_assert(DEPTH >= 2 && DEPTH <= 4, "");
#define ESLEEP 1
#define DOUT 4
static_assert(DOUT >= 2 && DOUT <= 4, "");
#define MSLEEP 6
#define PPOS 2
#define DLAST 2
static_assert(DLAST >= 2 && DLAST <= 4, "");
#define DSLEEP 4
constexpr int NR = 8;
constexpr int MM = 2048, NN = 2048, KSH = 256;
constexpr int PPD = 8;
constexpr int NPUSH = PPD * NR;
constexpr int NT = 6;
constexpr int NGEMM = 128;
constexpr int NDEC = PPD * NR;
constexpr int GRID = NPUSH + NGEMM + NDEC;
#define NCOMP 7
static_assert(NCOMP == 7, "");
static_assert(GRID <= 256, "all workgroups must be co-resident");

constexpr size_t XS_PER = 512u << 10;
constexpr size_t WS_PER = 256u << 10;
constexpr size_t XS_OFF = 0;
constexpr size_t WS_OFF = NR * XS_PER;
constexpr size_t OS_PER = 1024u << 10;
constexpr size_t OS_OFF = NR * (XS_PER + WS_PER);
constexpr size_t SLOT_BYTES = OS_OFF + NR * OS_PER;
constexpr size_t FLAG_OFF = SLOT_BYTES;
constexpr size_t FSTR = 128;
constexpr size_t WF_OFF = FLAG_OFF;
constexpr size_t XF_OFF = WF_OFF + 8 * 512;
constexpr size_t OF_OFF = XF_OFF + 64 * FSTR;
constexpr size_t TF_OFF = OF_OFF + 64 * 256;
constexpr size_t AF_OFF = TF_OFF;
constexpr size_t CNT_OFF = AF_OFF + 16 * FSTR;
constexpr size_t TMO_OFF = CNT_OFF + (size_t)GRID * FSTR;
constexpr size_t WS_NEEDED = TMO_OFF + FSTR;

constexpr u64 DEADLINE_TICKS = 40000000ull;

#define TS(i) do {} while (0)
#define TS_FLUSH() do {} while (0)

struct Params {
  const u16* x;
  const u16* w;
  char* myws;
  char* myout;
  int rank;
  u32 dmap;
  int pad_[2];
  char* ws[NR];
  char* out[NR];
};

constexpr int AUX_SYS = 17;
#define AUX_LD 17

__device__ __forceinline__ __amdgpu_buffer_rsrc_t mk_rsrc(const void* p, u32 bytes) {
  return __builtin_amdgcn_make_buffer_rsrc((void*)p, 0, bytes, 0x00020000);
}
__device__ __forceinline__ void st_sys(u32x4 v, __amdgpu_buffer_rsrc_t r, u32 off) {
  __builtin_amdgcn_raw_buffer_store_b128(v, r, off, 0, AUX_SYS);
}
__device__ __forceinline__ u32x4 ld_sys(__amdgpu_buffer_rsrc_t r, u32 off) {
  return __builtin_amdgcn_raw_buffer_load_b128(r, off, 0, AUX_LD);
}

typedef __attribute__((address_space(1))) u32 gu32;
__device__ __forceinline__ void st_flag(u32* f, u32 v) {
  __hip_atomic_store((gu32*)(size_t)f, v, __ATOMIC_RELAXED, __HIP_MEMORY_SCOPE_SYSTEM);
}

__device__ __forceinline__ bool wait_ge(const u32* f, u32 thr, u32 call, u32* tmo) {
  u64 t0 = 0;
  u32 spins = 0;
  for (;;) {
    const u32 v = __hip_atomic_load(f, __ATOMIC_RELAXED, __HIP_MEMORY_SCOPE_SYSTEM);
    if (__all(v >= thr)) break;
    if ((++spins & 31u) == 0) {
      const u64 now = __builtin_amdgcn_s_memrealtime();
      if (t0 == 0) t0 = now;
      if (__hip_atomic_load(tmo, __ATOMIC_RELAXED, __HIP_MEMORY_SCOPE_AGENT) == call ||
          now - t0 > DEADLINE_TICKS) {
        __hip_atomic_store(tmo, call, __ATOMIC_RELAXED, __HIP_MEMORY_SCOPE_AGENT);
        return false;
      }
    }
    __builtin_amdgcn_s_sleep(16);
  }
  asm volatile("" ::: "memory");
  return true;
}

__device__ __forceinline__ bool wait_flag(const u32* f, u32 call, u32* tmo) { return wait_ge(f, call, call, tmo); }

template <int N> __device__ __forceinline__ void wait_vm() {
  asm volatile("s_waitcnt vmcnt(%0)" :: "i"(N) : "memory");
}

__device__ __forceinline__ u32 pack_bf16(float a, float b) {
  u32 r;
  asm("v_cvt_pk_bf16_f32 %0, %1, %2" : "=v"(r) : "v"(a), "v"(b));
  return r;
}
__device__ __forceinline__ float silu(float y) {
  const float yc = fminf(fmaxf(y, -60.0f), 60.0f);
  return y / (1.0f + __expf(-yc));
}

constexpr int PROW = 272;
constexpr int PART_BYTES = 16 * PROW;
constexpr int PBUF = 8 * PART_BYTES + 64;
constexpr int LDS_BYTES = 2 * PBUF;

constexpr bool round_is_w(int n) { return n < 2; }

constexpr int pace_cnt(int j) {
  int x = DEPTH - 1;
  for (int n = 0; n + 2 < NT; ++n)
    if (j - DEPTH <= 4 * n + 3 && 4 * n + 3 <= j - 1) x += 4;
  return x;
}
template <int I> struct IC { static constexpr int value = I; };
template <int N, int I = 0, class F> __device__ __forceinline__ void static_for(F&& f) {
  if constexpr (I < N) { f(IC<I>{}); static_for<N, I + 1>(f); }
}

struct Unit { u32x4 L; u32x2 N, E; u32 info; };
__device__ __forceinline__ u32 pkmax16(u32 a, u32 b) {
  u32 r; asm("v_pk_max_u16 %0, %1, %2" : "=v"(r) : "v"(a), "v"(b)); return r;
}
__device__ __forceinline__ u32 wave_umax(u32 t) {
  t = max(t, (u32)__builtin_amdgcn_update_dpp(0, (int)t, 0x111, 0xf, 0xf, false));
  t = max(t, (u32)__builtin_amdgcn_update_dpp(0, (int)t, 0x112, 0xf, 0xf, false));
  t = max(t, (u32)__builtin_amdgcn_update_dpp(0, (int)t, 0x114, 0xf, 0xf, false));
  t = max(t, (u32)__builtin_amdgcn_update_dpp(0, (int)t, 0x118, 0xf, 0xf, false));
  t = max(t, (u32)__builtin_amdgcn_update_dpp(0, (int)t, 0x142, 0xa, 0xf, false));
  t = max(t, (u32)__builtin_amdgcn_update_dpp(0, (int)t, 0x143, 0xc, 0xf, false));
  return (u32)__builtin_amdgcn_readlane((int)t, 63);
}
__device__ __forceinline__ u32x4 enc_low(const u32* d) {
  return u32x4{__builtin_amdgcn_perm(d[1], d[0], 0x06040200u), __builtin_amdgcn_perm(d[3], d[2], 0x06040200u),
               __builtin_amdgcn_perm(d[5], d[4], 0x06040200u), __builtin_amdgcn_perm(d[7], d[6], 0x06040200u)};
}
__device__ __forceinline__ void enc_h(const u32 (&h)[4], Unit& u) {
  u32 x[4], t[4], q[4];
#pragma unroll
  for (int i = 0; i < 4; ++i) x[i] = h[i] & 0x7f7f7f7fu;
  const u32 m = pkmax16(pkmax16(x[0], x[1]), pkmax16(x[2], x[3]));
  const u32 emax = wave_umax(max(m, m << 16)) >> 24;
  const u32 C = emax < 7u ? 0u : emax - 7u;
  const u32 K = (128u - C) * 0x01010101u;
#pragma unroll
  for (int i = 0; i < 4; ++i) {
    t[i] = x[i] + K;
    q[i] = (h[i] & 0x80808080u) | (t[i] & 0x07070707u);
  }
  u.N.x = q[0] | __builtin_amdgcn_alignbit(q[1], q[1], 4);
  u.N.y = q[2] | __builtin_amdgcn_alignbit(q[3], q[3], 4);
  u.E.x = ((t[0] >> 3) & 0x0f0f0f0fu) | ((t[1] << 1) & 0xf0f0f0f0u);
  u.E.y = ((t[2] >> 3) & 0x0f0f0f0fu) | ((t[3] << 1) & 0xf0f0f0f0u);
  const u64 nz = __ballot((u.E.x | u.E.y) != 0u);
  u32 esc = 0;
#pragma unroll
  for (int i = 0; i < 4; ++i) esc |= ((nz >> (16 * i)) & 0xffffull) ? (1u << i) : 0u;
  u.info = C | (esc << 7);
}
__device__ __forceinline__ void enc_high(const u32* d, Unit& u) {
  u32 h[4];
#pragma unroll
  for (int i = 0; i < 4; ++i) h[i] = __builtin_amdgcn_perm(d[2 * i + 1], d[2 * i], 0x07050301u);
  enc_h(h, u);
}
__device__ __forceinline__ void dec_unit(u32x4 L, u32x2 N, u32x2 E, u32 inf, int lane, u32x4& a, u32x4& b) {
  const u32 Cv = (inf & 127u) * 0x01010101u;
  u32 h0 = (N.x & 0x87878787u) + Cv, h1 = (__builtin_amdgcn_alignbit(N.x, N.x, 28) & 0x87878787u) + Cv;
  u32 h2 = (N.y & 0x87878787u) + Cv, h3 = (__builtin_amdgcn_alignbit(N.y, N.y, 28) & 0x87878787u) + Cv;
  if (__builtin_expect(inf >> 7, 0)) {
    if (!((inf >> (7 + (lane >> 4))) & 1u)) E = u32x2{0u, 0u};
    h0 = (h0 & 0x80808080u) | (((h0 & 0x7f7f7f7fu) + ((E.x & 0x0f0f0f0fu) << 3)) & 0x7f7f7f7fu);
    h1 = (h1 & 0x80808080u) | (((h1 & 0x7f7f7f7fu) + ((E.x & 0xf0f0f0f0u) >> 1)) & 0x7f7f7f7fu);
    h2 = (h2 & 0x80808080u) | (((h2 & 0x7f7f7f7fu) + ((E.y & 0x0f0f0f0fu) << 3)) & 0x7f7f7f7fu);
    h3 = (h3 & 0x80808080u) | (((h3 & 0x7f7f7f7fu) + ((E.y & 0xf0f0f0f0u) >> 1)) & 0x7f7f7f7fu);
  }
  a = u32x4{__builtin_amdgcn_perm(h0, L.x, 0x05010400u), __builtin_amdgcn_perm(h0, L.x, 0x07030602u),
            __builtin_amdgcn_perm(h1, L.y, 0x05010400u), __builtin_amdgcn_perm(h1, L.y, 0x07030602u)};
  b = u32x4{__builtin_amdgcn_perm(h2, L.z, 0x05010400u), __builtin_amdgcn_perm(h2, L.z, 0x07030602u),
            __builtin_amdgcn_perm(h3, L.w, 0x05010400u), __builtin_amdgcn_perm(h3, L.w, 0x07030602u)};
}

__device__ __forceinline__ void split_unit(const u32x4& a, const u32x4& b, u32x4& l, u32 (&h)[4]) {
  asm volatile("v_perm_b32 %0, %9, %8, %16\n\t"
               "v_perm_b32 %1, %11, %10, %16\n\t"
               "v_perm_b32 %2, %13, %12, %16\n\t"
               "v_perm_b32 %3, %15, %14, %16\n\t"
               "v_perm_b32 %4, %9, %8, %17\n\t"
               "v_perm_b32 %5, %11, %10, %17\n\t"
               "v_perm_b32 %6, %13, %12, %17\n\t"
               "v_perm_b32 %7, %15, %14, %17"
               : "=&v"(l.x), "=&v"(l.y), "=&v"(l.z), "=&v"(l.w), "=&v"(h[0]), "=&v"(h[1]), "=&v"(h[2]), "=&v"(h[3])
               : "v"(a.x), "v"(a.y), "v"(a.z), "v"(a.w), "v"(b.x), "v"(b.y), "v"(b.z), "v"(b.w),
                 "s"(0x06040200u), "s"(0x07050301u)
               : "memory");
}

constexpr u32 SENT = 0xffffffffu;
template <int N>
__device__ __forceinline__ bool valid4(u32x4 (&v)[4]) {
  u64 m; u32 t;
  asm volatile("s_waitcnt vmcnt(%18)\n\t"
               "v_max3_u32 %1, %2, %3, %4\n\t"
               "v_max3_u32 %1, %1, %5, %6\n\t"
               "v_max3_u32 %1, %1, %7, %8\n\t"
               "v_max3_u32 %1, %1, %9, %10\n\t"
               "v_max3_u32 %1, %1, %11, %12\n\t"
               "v_max3_u32 %1, %1, %13, %14\n\t"
               "v_max3_u32 %1, %1, %15, %16\n\t"
               "v_max_u32 %1, %1, %17\n\t"
               "v_cmp_eq_u32_e64 %0, -1, %1\n\t"
               "s_nop 4"
               : "=s"(m), "=&v"(t)
               : "v"(v[0].x), "v"(v[0].y), "v"(v[0].z), "v"(v[0].w), "v"(v[1].x), "v"(v[1].y), "v"(v[1].z), "v"(v[1].w),
                 "v"(v[2].x), "v"(v[2].y), "v"(v[2].z), "v"(v[2].w), "v"(v[3].x), "v"(v[3].y), "v"(v[3].z), "v"(v[3].w),
                 "i"(N)
               : "memory");
  return m == 0ull;
}

__global__ void __launch_bounds__(512, 2) gemm_ar_silu(Params p) {
  __shared__ __attribute__((aligned(16))) char smem[LDS_BYTES];
  const int tid = threadIdx.x, lane = tid & 63;
  const int wv = __builtin_amdgcn_readfirstlane(tid >> 6);
  const int bid = blockIdx.x;
  const bool is_engine = bid < NPUSH;
  const int xs = bid & 7;
  const int sub = bid >> 3;
#define GMAP 1
  const int gs = (bid - NPUSH) >> 3;
  const int g = ((xs & ((8 >> GMAP) - 1)) << GMAP) + (gs & ((1 << GMAP) - 1));
  const int r16 = (xs >> (3 - GMAP)) * (16 >> GMAP) + (gs >> GMAP);
  u32 dmap = p.dmap;
  int rank = p.rank;
  u64 a0 = (u64)p.myws, a1 = (u64)p.myout, a2 = (u64)p.x, a3 = (u64)p.w, a4 = (u64)p.ws[xs], a5 = (u64)p.out[xs];
  asm volatile("" : "+s"(rank), "+s"(dmap), "+s"(a0), "+s"(a1), "+s"(a2), "+s"(a3), "+s"(a4), "+s"(a5));
  const int dst = (dmap >> (4 * xs)) & 7;
  typedef __attribute__((address_space(1))) char gchar;
  char* const myws = (char*)(gchar*)a0;
  char* const myout = (char*)(gchar*)a1;
  const u16* const px = (const u16*)(char*)(gchar*)a2;
  const u16* const pw = (const u16*)(char*)(gchar*)a3;
  char* const dws = (char*)(gchar*)a4;
  char* const dout = (char*)(gchar*)a5;
  u32* cntp = (u32*)(myws + CNT_OFF + (size_t)bid * FSTR);
  u32* tmo = (u32*)(myws + TMO_OFF);
  TS(0);

  if (__builtin_expect(is_engine, 1)) {
    if (wv > 5) return;
    const int di = dst >> 2, dj = dst & 3;
    u32* lcnt = (u32*)smem;
    if (wv >= 4) {
      const u32 cnt = __hip_atomic_load(cntp, __ATOMIC_RELAXED, __HIP_MEMORY_SCOPE_AGENT);
      __builtin_amdgcn_s_setprio(3);
      const u32 call = __builtin_amdgcn_readfirstlane(cnt) + 1u;
      if (wv == 4) lcnt[lane] = lane == 16 ? call : 0u;
      asm volatile("s_waitcnt lgkmcnt(0)" :: "s"(call) : "memory");
      __builtin_amdgcn_s_barrier();
      asm volatile("" ::: "memory");
      if (wv >= 5) {
        if (dst == rank) return;
        {
          const __amdgpu_buffer_rsrc_t mrs = mk_rsrc(myout, (u32)(MM * NN * 2));
          const __amdgpu_buffer_rsrc_t srs6 = mk_rsrc(myws + OS_OFF + (size_t)dst * OS_PER, (u32)OS_PER);
          const u32 eo = (u32)(((1024 * di + 16 * (49 + 2 * sub)) * NN + 512 * dj) * 2 + lane * 16);
#pragma unroll 1
          for (int k = 0; k < 16; ++k) {
            st_sys(u32x4{SENT, SENT, SENT, SENT}, srs6, (u32)((48 + sub) * 16384 + k * 1024 + lane * 16));
            __builtin_amdgcn_s_sleep(MSLEEP);
            st_sys(u32x4{SENT, SENT, SENT, SENT}, mrs, eo + k * NN * 2);
            __builtin_amdgcn_s_sleep(MSLEEP);
          }
          wait_vm<0>();
          if (lane == 0) __hip_atomic_store(lcnt + 14, 1u, __ATOMIC_RELAXED, __HIP_MEMORY_SCOPE_WORKGROUP);
        }
        bool good = wait_flag((const u32*)(myws + AF_OFF + (size_t)sub * 2 * FSTR) + lane, call, tmo);
        if (good && lane == 0) __hip_atomic_store(lcnt + 12, 2u, __ATOMIC_RELAXED, __HIP_MEMORY_SCOPE_WORKGROUP);
        TS(9);
        const u64 t0 = __builtin_amdgcn_s_memrealtime();
        u32 spins = 0;
        while (good && __hip_atomic_load(lcnt + 15, __ATOMIC_RELAXED, __HIP_MEMORY_SCOPE_WORKGROUP) < 4u) {
          __builtin_amdgcn_s_sleep(8);
          if ((++spins & 63u) == 0 &&
              (__hip_atomic_load(tmo, __ATOMIC_RELAXED, __HIP_MEMORY_SCOPE_AGENT) == call ||
               __builtin_amdgcn_s_memrealtime() - t0 > DEADLINE_TICKS)) good = false;
        }
        if (!good) {
          __hip_atomic_store(tmo, call, __ATOMIC_RELAXED, __HIP_MEMORY_SCOPE_AGENT);
          if (lane == 0) {
            __hip_atomic_store(lcnt + 13, 1u, __ATOMIC_RELAXED, __HIP_MEMORY_SCOPE_WORKGROUP);
            asm volatile("s_waitcnt lgkmcnt(0)" ::: "memory");
            __hip_atomic_store(lcnt + 12, 1u, __ATOMIC_RELAXED, __HIP_MEMORY_SCOPE_WORKGROUP);
          }
        }
        TS(10);
        TS_FLUSH();
        return;
      }
      const __amdgpu_buffer_rsrc_t frs = mk_rsrc(dws + FLAG_OFF, (u32)(TF_OFF - FLAG_OFF));
#pragma unroll
      for (int h = 0; h < 2 * NT; ++h) {
        u32 fo;
        if (h < 4) {
          fo = (u32)(WF_OFF - FLAG_OFF) + (u32)(sub * 512 + (rank * 4 + h) * 16);
        } else {
          const int rfi = 2 * (8 * ((h - 4) >> 1) + sub) + (h & 1);
          fo = (u32)(XF_OFF - FLAG_OFF) + (u32)(rfi * 128 + rank * 16);
        }
        while (__hip_atomic_load(lcnt + h, __ATOMIC_RELAXED, __HIP_MEMORY_SCOPE_WORKGROUP) < 4u)
          __builtin_amdgcn_s_sleep(1);
        if (h == 2 * NT - 2 && dst != rank)
          while (__hip_atomic_load(lcnt + 14, __ATOMIC_RELAXED, __HIP_MEMORY_SCOPE_WORKGROUP) == 0u)
            __builtin_amdgcn_s_sleep(1);
        asm volatile("" ::: "memory");
        if (lane < 2) {
          const u32 inf = __hip_atomic_load(lcnt + 32 + 2 * h + lane, __ATOMIC_RELAXED, __HIP_MEMORY_SCOPE_WORKGROUP);
          __builtin_amdgcn_raw_buffer_store_b64(u32x2{call, inf | (call << 22)}, frs, fo + lane * 8, 0, AUX_SYS);
        }
        if (h & 1) TS(2 + (h >> 1));
      }
      if (dst != rank) {
        const u32 ofo = (u32)(OF_OFF - FLAG_OFF) + (u32)((rank * 8 + sub) * 256);
#pragma unroll 1
        for (int m = 0; m < NCOMP; ++m) {
          while (__hip_atomic_load(lcnt + 20 + m, __ATOMIC_RELAXED, __HIP_MEMORY_SCOPE_WORKGROUP) < 4u)
            __builtin_amdgcn_s_sleep(1);
          asm volatile("" ::: "memory");
          if (lane < 4) {
            const u32 inf = __hip_atomic_load(lcnt + 64 + 4 * m + lane, __ATOMIC_RELAXED, __HIP_MEMORY_SCOPE_WORKGROUP);
            if (m == 5) {
              const u32 e6 = __hip_atomic_load(lcnt + 96 + lane, __ATOMIC_RELAXED, __HIP_MEMORY_SCOPE_WORKGROUP);
              st_sys(u32x4{call, inf | (call << 22), (e6 >> 31) ? call : 0u, (e6 & 0x3fffffu) | (call << 22)}, frs,
                     ofo + 192 + lane * 16);
            } else {
              __builtin_amdgcn_raw_buffer_store_b64(u32x2{call, inf | (call << 22)}, frs,
                                                    ofo + (u32)(min(m, 5) * 32 + lane * 8), 0, AUX_SYS);
            }
          }
        }
      }
      if (lane == 0) __hip_atomic_store(cntp, call, __ATOMIC_RELAXED, __HIP_MEMORY_SCOPE_AGENT);
      TS(15);
      TS_FLUSH();
      return;
    }
    const int c = lane & 15, kg = lane >> 4;
    u32 r[NT][16];
    auto load_task = [&](int n) {
      if (round_is_w(n)) {
        const int kh = n, g = sub, ks = 4 * kh + wv;
        const u16* src = pw + (size_t)(32 * ks + 8 * kg) * NN + 512 * dj + 64 * g + 4 * c;
#pragma unroll
        for (int jj = 0; jj < 8; ++jj) {
          const u32x2 v = *(const u32x2*)(src + (size_t)jj * NN);
          r[n][2 * jj] = v.x; r[n][2 * jj + 1] = v.y;
        }
      } else {
        const int rfi = 2 * (8 * (n - 2) + sub);
        const u16* src = px + (size_t)(1024 * di + 16 * rfi + c) * KSH + 64 * wv + 8 * kg;
#pragma unroll
        for (int k = 0; k < 4; ++k) {
          const u32x4 v = *(const u32x4*)(src + (k >> 1) * 16 * KSH + 32 * (k & 1));
          r[n][4 * k] = v.x; r[n][4 * k + 1] = v.y; r[n][4 * k + 2] = v.z; r[n][4 * k + 3] = v.w;
        }
      }
    };
    auto transpose = [&](int n) {
      u32 f[16];
#pragma unroll
      for (int d = 0; d < 4; ++d) {
        const u32 a0 = r[n][4 * d], a1 = r[n][4 * d + 1], b0 = r[n][4 * d + 2], b1 = r[n][4 * d + 3];
        f[0 + d] = __builtin_amdgcn_perm(b0, a0, 0x05040100u);
        f[4 + d] = __builtin_amdgcn_perm(b0, a0, 0x07060302u);
        f[8 + d] = __builtin_amdgcn_perm(b1, a1, 0x05040100u);
        f[12 + d] = __builtin_amdgcn_perm(b1, a1, 0x07060302u);
      }
#pragma unroll
      for (int e = 0; e < 16; ++e) r[n][e] = f[e];
    };
    const __amdgpu_buffer_rsrc_t wrs = mk_rsrc(dws + WS_OFF + (size_t)rank * WS_PER, (u32)WS_PER);
    const __amdgpu_buffer_rsrc_t xrs = mk_rsrc(dws + XS_OFF + (size_t)rank * XS_PER, (u32)XS_PER);
    auto slot = [&](int h) -> u32 {
      const int n = h >> 1, hf = h & 1;
      if (round_is_w(n)) return (u32)(((sub * 8 + 4 * n + wv) * 4 + 2 * hf) * 1024);
      return (u32)(((2 * (8 * (n - 2) + sub) + hf) * 8 + 2 * wv) * 1024);
    };
    u32 info[2 * NT];
    auto send_esc = [&](const Unit& u, const __amdgpu_buffer_rsrc_t& rs, u32 so) {
      if ((u.info >> (7 + (lane >> 4))) & 1u)
        __builtin_amdgcn_raw_buffer_store_b64(u.E, rs, so + 1536 + lane * 8, 0, AUX_SYS);
    };
    auto signal_h = [&](int h) {
      if (lane == 0) {
        __hip_atomic_fetch_or(lcnt + 32 + 2 * h + (wv >> 1), info[h] << (11 * (wv & 1)), __ATOMIC_RELAXED, __HIP_MEMORY_SCOPE_WORKGROUP);
        __hip_atomic_fetch_add(lcnt + h, 1u, __ATOMIC_RELAXED, __HIP_MEMORY_SCOPE_WORKGROUP);
      }
    };
    constexpr int HS = (4 * (NT - 1) - DEPTH) / 2;
    load_task(0);
    __builtin_amdgcn_sched_barrier(0);
    load_task(1);
    __builtin_amdgcn_sched_barrier(0);
    TS(1);
    transpose(0);
    Unit cu;
    static_for<4 * (NT - 1)>([&](auto J) {
      constexpr int j = decltype(J)::value;
      constexpr int h = j >> 1;
      if (j == DEPTH) {
        transpose(1);
        __builtin_amdgcn_s_barrier();
        asm volatile("" ::: "memory");
      }
      if ((j & 1) == 0) cu.L = enc_low(&r[h >> 1][8 * (h & 1)]);
      else { enc_high(&r[h >> 1][8 * (h & 1)], cu); info[h] = cu.info; send_esc(cu, h < 4 ? wrs : xrs, slot(h)); }
      if (j >= DEPTH) {
        wait_vm<pace_cnt(j)>();
        if (((j - DEPTH) & 1) == 1) signal_h((j - DEPTH) >> 1);
        if (((j - DEPTH) & 3) == 3) TS(2 + ((j - DEPTH) >> 2));
      }
      if ((j & 1) == 0) st_sys(cu.L, h < 4 ? wrs : xrs, slot(h) + lane * 16);
      else __builtin_amdgcn_raw_buffer_store_b64(cu.N, h < 4 ? wrs : xrs, slot(h) + 1024 + lane * 8, 0, AUX_SYS);
      __builtin_amdgcn_sched_barrier(0);
      if ((j & 3) == 3 && (j >> 2) + 2 < NT) { load_task((j >> 2) + 2); __builtin_amdgcn_sched_barrier(0); }
    });
    int pk = 3, pm = 0;
    u32 pinfo = 0, early = 0;
    auto flush = [&]() {
      if (pk == 3) {
#pragma unroll
        for (int h = HS; h < 2 * NT - 2; ++h) signal_h(h);
        TS(2 + NT - 2);
      }
      if (pk == 1) { signal_h(2 * NT - 2); signal_h(2 * NT - 1); TS(2 + NT - 1); }
      if (pk == 2 && lane == 0) {
        __hip_atomic_store(lcnt + 64 + 4 * pm + wv, pinfo, __ATOMIC_RELAXED, __HIP_MEMORY_SCOPE_WORKGROUP);
        if (pm == 5) __hip_atomic_store(lcnt + 96 + wv, early, __ATOMIC_RELAXED, __HIP_MEMORY_SCOPE_WORKGROUP);
        __hip_atomic_fetch_add(lcnt + 20 + pm, 1u, __ATOMIC_RELAXED, __HIP_MEMORY_SCOPE_WORKGROUP);
      }
      pk = 0;
    };
    const int ri = rank >> 2, rj = rank & 3;
    const u32x4 srs = {(u32)a1, (u32)(a1 >> 32) & 0xffffu, (u32)(MM * NN * 2), 0x00020000u};
    const u32x4 drs = {(u32)a5, (u32)(a5 >> 32) & 0xffffu, (u32)(MM * NN * 2), 0x00020000u};
    const u32 off0 = (u32)(((1024 * ri + 32 * sub + 4 * wv) * NN + 512 * rj) * 2 + lane * 16);
    auto moff = [&](int m) -> u32 { return (u32)(16 * (m >> 1) + (m & 1)) * (u32)(16 * NN * 2); };
    u32 lst = 0;
    auto ld4 = [&](int m, u32x4 (&v)[4]) {
      lst = __builtin_amdgcn_readfirstlane(__hip_atomic_load(lcnt + 12, __ATOMIC_RELAXED, __HIP_MEMORY_SCOPE_WORKGROUP));
      const u32 so = moff(m);
      asm volatile("buffer_load_dwordx4 %0, %4, %5, %6 offen sc0 sc1\n\t"
                   "buffer_load_dwordx4 %1, %4, %5, %7 offen sc0 sc1\n\t"
                   "buffer_load_dwordx4 %2, %4, %5, %8 offen sc0 sc1\n\t"
                   "buffer_load_dwordx4 %3, %4, %5, %9 offen sc0 sc1"
                   : "=&v"(v[0]), "=&v"(v[1]), "=&v"(v[2]), "=&v"(v[3])
                   : "v"(off0), "s"(srs), "s"(so), "s"(so + NN * 2), "s"(so + 2 * NN * 2), "s"(so + 3 * NN * 2)
                   : "memory");
    };
    auto task4 = [&](auto PRE, int mn, u32x4 (&nxt)[4], auto&& st) {
      constexpr bool pre = decltype(PRE)::value & 1;
      constexpr int DD = (decltype(PRE)::value & 2) ? DEPTH : (decltype(PRE)::value & 1) ? DOUT : DLAST;
      static_for<4>([&](auto K) {
        constexpr int k = decltype(K)::value;
        if (pre && k == PPOS) ld4(mn, nxt);
        wait_vm<DD - 1 + ((pre && PPOS <= k && k < PPOS + DD) ? 4 : 0)>();
        if (k == DD - 1) flush();
        st(K);
        asm volatile("" ::: "memory");
      });
    };
    auto send_units = [&](auto PRE, int mn, u32x4 (&nxt)[4], const Unit& u0, const Unit& u1,
                          const __amdgpu_buffer_rsrc_t& rs, u32 so0, u32 so1) {
      send_esc(u0, rs, so0);
      send_esc(u1, rs, so1);
      task4(PRE, mn, nxt, [&](auto K) {
        constexpr int k = decltype(K)::value;
        if (k == 0) st_sys(u0.L, rs, so0 + lane * 16);
        if (k == 1) __builtin_amdgcn_raw_buffer_store_b64(u0.N, rs, so0 + 1024 + lane * 8, 0, AUX_SYS);
        if (k == 2) st_sys(u1.L, rs, so1 + lane * 16);
        if (k == 3) __builtin_amdgcn_raw_buffer_store_b64(u1.N, rs, so1 + 1024 + lane * 8, 0, AUX_SYS);
      });
    };
    u32x4 A[4], B[4];
    {
      Unit u0, u1;
      u0.L = enc_low(&r[NT - 1][0]); enc_high(&r[NT - 1][0], u0); info[2 * NT - 2] = u0.info;
      u1.L = enc_low(&r[NT - 1][8]); enc_high(&r[NT - 1][8], u1); info[2 * NT - 1] = u1.info;
      asm volatile("" : "+v"(u0.L), "+v"(u0.N), "+v"(u1.L), "+v"(u1.N) :: "memory");
      send_units(IC<3>{}, 0, A, u0, u1, xrs, slot(2 * NT - 2), slot(2 * NT - 1));
      pk = 1;
      if (dst == rank) { wait_vm<0>(); flush(); }
    }
    if (dst != rank) {
      const __amdgpu_buffer_rsrc_t osr = mk_rsrc(dws + OS_OFF + (size_t)rank * OS_PER, (u32)OS_PER);
      auto settle = [&](int m, u32x4 (&cur)[4]) {
        bool okv = valid4<4 - PPOS>(cur);
        if (!((lst & 1u) || ((lst & 2u) && okv))) {
          for (;;) {
            ld4(m, cur);
            wait_vm<4>();
            if (m != 6) flush();
            okv = valid4<0>(cur);
            if ((lst & 1u) || ((lst & 2u) && okv)) break;
            __builtin_amdgcn_s_sleep(DSLEEP);
          }
        }
      };
      auto raw1 = [&](const u32x4& v, u32 so) {
        asm volatile("buffer_store_dwordx4 %0, %1, %2, %3 offen sc0 sc1\n\ts_nop 1"
                     :: "v"(v), "v"(off0), "s"(drs), "s"(so) : "memory");
      };
      auto step = [&](auto COMP, int m, u32x4 (&cur)[4], u32x4 (&nxt)[4]) {
        settle(m, cur);
        if constexpr (decltype(COMP)::value) {
          Unit u0, u1;
          u32 h0[4], h1[4];
          split_unit(cur[0], cur[1], u0.L, h0);
          split_unit(cur[2], cur[3], u1.L, h1);
          enc_h(h0, u0);
          enc_h(h1, u1);
          const u32 so = (u32)(((m * 8 + sub) * 4 + wv) * 4096);
          if (decltype(COMP)::value == 2) early = u0.info | (u1.info << 11) | 0x80000000u;
          send_units(IC<1>{}, m + 1, nxt, u0, u1, osr, so, so + 2048);
          pk = 2; pm = m; pinfo = u0.info | (u1.info << 11);
        } else {
          const u32 so = moff(m);
          task4(IC<1>{}, m + 1, nxt, [&](auto K) { raw1(cur[decltype(K)::value], so + decltype(K)::value * NN * 2); });
        }
      };
      TS(8);
#pragma unroll 1
      for (int m = 0; m < 6; m += 2) { step(IC<1>{}, m, A, B); step(IC<1>{}, m + 1, B, A); }
      step(IC<2>{}, 6, A, B);
      {
        settle(7, B);
        if (lane == 0) __hip_atomic_fetch_add(lcnt + 15, 1u, __ATOMIC_RELAXED, __HIP_MEMORY_SCOPE_WORKGROUP);
        const bool bad = __hip_atomic_load(lcnt + 13, __ATOMIC_RELAXED, __HIP_MEMORY_SCOPE_WORKGROUP) != 0u;
        const u32 so = moff(7);
        if (__builtin_expect(!bad, 1)) {
          task4(IC<0>{}, 0, A, [&](auto K) {
            constexpr int k = decltype(K)::value;
            if (k == DLAST - 1) TS(13);
            raw1(B[k], so + k * NN * 2);
          });
        } else {
          wait_vm<0>();
          flush();
          const __amdgpu_buffer_rsrc_t prs = mk_rsrc(dout, (u32)(MM * NN * 2));
          for (int m = NCOMP; m < 8; ++m)
            for (int k = 0; k < 4; ++k)
              st_sys(u32x4{0x7fc07fc0u, 0x7fc07fc0u, 0x7fc07fc0u, 0x7fc07fc0u}, prs, off0 + moff(m) + k * NN * 2);
          wait_vm<0>();
        }
      }
      const u32 call = __builtin_amdgcn_readfirstlane(lcnt[16]);
      const __amdgpu_buffer_rsrc_t mrs = mk_rsrc(myout, (u32)(MM * NN * 2));
      const u32 eo = (u32)(((1024 * di + 16 * (49 + 2 * sub) + 4 * wv) * NN + 512 * dj) * 2 + lane * 16);
      u64 t0 = 0; u32 spins = 0;
      for (;;) {
        u32 mx = 0;
#pragma unroll
        for (int k = 0; k < 4; ++k) {
          const u32x4 v = __builtin_amdgcn_raw_buffer_load_b128(mrs, eo + k * NN * 2, 0, AUX_SYS);
          mx = max(max(mx, v.x), max(v.y, max(v.z, v.w)));
        }
        if (__all(mx != SENT)) break;
        if ((++spins & 31u) == 0) {
          const u64 now = __builtin_amdgcn_s_memrealtime();
          if (t0 == 0) t0 = now;
          if (__hip_atomic_load(tmo, __ATOMIC_RELAXED, __HIP_MEMORY_SCOPE_AGENT) == call || now - t0 > DEADLINE_TICKS) {
            __hip_atomic_store(tmo, call, __ATOMIC_RELAXED, __HIP_MEMORY_SCOPE_AGENT);
            break;
          }
        }
        __builtin_amdgcn_s_sleep(ESLEEP);
        asm volatile("" ::: "memory");
      }
      TS(12);
    }
    TS_FLUSH();
    return;
  }

  if (bid >= NPUSH + NGEMM) {
    if (dst == rank) return;
    const int dsub = (bid - NPUSH - NGEMM) >> 3;
    const u32 cnt = __hip_atomic_load(cntp, __ATOMIC_RELAXED, __HIP_MEMORY_SCOPE_AGENT);
    const u32 call = __builtin_amdgcn_readfirstlane(cnt) + 1u;
    __builtin_amdgcn_s_barrier();
    if (tid == 0) __hip_atomic_store(cntp, call, __ATOMIC_RELAXED, __HIP_MEMORY_SCOPE_AGENT);
    const int dw = wv & 3, du = wv >> 2;
    const __amdgpu_buffer_rsrc_t fr = mk_rsrc(myws + FLAG_OFF, (u32)(TF_OFF - FLAG_OFF));
    const __amdgpu_buffer_rsrc_t sr = mk_rsrc(myws + OS_OFF + (size_t)dst * OS_PER, (u32)OS_PER);
    const __amdgpu_buffer_rsrc_t orr = mk_rsrc(myout, (u32)(MM * NN * 2));
    const u32 oo = (u32)(((1024 * (dst >> 2) + 32 * dsub + 4 * dw + 2 * du) * NN + 512 * (dst & 3)) * 2 + lane * 16);
    __builtin_amdgcn_s_sleep(127);
    __builtin_amdgcn_s_sleep(127);
    const u32 fo0 = (u32)(OF_OFF - FLAG_OFF) + (u32)((dst * 8 + dsub) * 256);
    u64 t0 = 0; u32 spins = 0; bool ok = true;
    auto expired = [&]() -> bool {
      if ((++spins & 31u) == 0) {
        const u64 now = __builtin_amdgcn_s_memrealtime();
        if (t0 == 0) t0 = now;
        if (__builtin_amdgcn_readfirstlane(__hip_atomic_load(tmo, __ATOMIC_RELAXED, __HIP_MEMORY_SCOPE_AGENT)) == call ||
            now - t0 > DEADLINE_TICKS) {
          __hip_atomic_store(tmo, call, __ATOMIC_RELAXED, __HIP_MEMORY_SCOPE_AGENT);
          ok = false;
        }
      }
      return !ok;
    };
    struct Pl { u32x4 L; u32x2 N, E; };
    auto fetch1 = [&](int m, u32 inf, Pl& P) {
      const u32 so = (u32)((((m * 8 + dsub) * 4 + dw) * 2 + du) * 2048);
      P.L = __builtin_amdgcn_raw_buffer_load_b128(sr, so + lane * 16, 0, AUX_SYS);
      P.N = __builtin_amdgcn_raw_buffer_load_b64(sr, so + 1024 + lane * 8, 0, AUX_SYS);
      P.E = u32x2{0u, 0u};
      if (inf >> 7) P.E = __builtin_amdgcn_raw_buffer_load_b64(sr, so + 1536 + lane * 8, 0, AUX_SYS);
    };
    auto put = [&](int m, u32 inf, const Pl& P) {
      u32x4 a, b;
      dec_unit(P.L, P.N, P.E, inf, lane, a, b);
      if (!ok) { a = u32x4{0x7fc07fc0u, 0x7fc07fc0u, 0x7fc07fc0u, 0x7fc07fc0u}; b = a; }
      const u32 ro = oo + (u32)(16 * (m >> 1) + (m & 1)) * (u32)(16 * NN * 2);
      st_sys(a, orr, ro);
      st_sys(b, orr, ro + NN * 2);
      TS(1 + m);
    };
    auto clean = [&](u32 inf, Pl& P) -> bool {
      if (!((inf >> (7 + (lane >> 4))) & 1u)) P.E = u32x2{0u, 0u};
      const u32 mx = max(max(max(P.L.x, P.L.y), max(P.L.z, P.L.w)), max(max(P.N.x, P.N.y), max(P.E.x, P.E.y)));
      return __all(mx != SENT);
    };
    u32 inf6 = 0; bool early = false, done = false;
    Pl P6;
#pragma unroll 1
    for (int m = 0; m < 6; ++m) {
      const u32 fo = fo0 + (u32)(m < 5 ? m * 32 + dw * 8 : 192 + dw * 16);
      u32 inf;
      for (;;) {
        const u32x4 v = __builtin_amdgcn_raw_buffer_load_b128(fr, fo & ~15u, 0, AUX_SYS);
        const bool hi = (fo & 8u) != 0u;
        inf = __builtin_amdgcn_readfirstlane(hi ? v.w : v.y);
        inf6 = __builtin_amdgcn_readfirstlane(v.w);
        early = m == 5 && __builtin_amdgcn_readfirstlane(v.z) == call && (inf6 >> 22) == (call & 1023u);
        if (__builtin_amdgcn_readfirstlane(hi ? v.z : v.x) == call && (inf >> 22) == (call & 1023u)) break;
        if (expired()) break;
        __builtin_amdgcn_s_sleep(PSLEEP);
        asm volatile("" ::: "memory");
      }
      asm volatile("" ::: "memory");
      inf = (inf >> (11 * du)) & 0x7ffu;
      inf6 = (inf6 >> (11 * du)) & 0x7ffu;
      Pl P;
      fetch1(m, inf, P);
      if (early) {
        fetch1(6, inf6, P6);
        done = clean(inf6, P6);
      }
      put(m, inf, P);
    }
    while (!done) {
      const u32x2 v = __builtin_amdgcn_raw_buffer_load_b64(fr, fo0 + (u32)(160 + dw * 8), 0, AUX_SYS);
      if (early) {
        fetch1(6, inf6, P6);
        if (clean(inf6, P6)) break;
      }
      const u32 fi = __builtin_amdgcn_readfirstlane(v.y);
      if ((__builtin_amdgcn_readfirstlane(v.x) == call && (fi >> 22) == (call & 1023u)) || expired()) {
        asm volatile("" ::: "memory");
        inf6 = (fi >> (11 * du)) & 0x7ffu;
        fetch1(6, inf6, P6);
        break;
      }
      __builtin_amdgcn_s_sleep(ESLEEP);
      asm volatile("" ::: "memory");
    }
    put(6, inf6, P6);
    TS_FLUSH();
    return;
  }

  const int ri = rank >> 2, rj = rank & 3;
  const u32 cnt = __hip_atomic_load(cntp, __ATOMIC_RELAXED, __HIP_MEMORY_SCOPE_AGENT);
  const u32 call = __builtin_amdgcn_readfirstlane(cnt) + 1u;
  const int q = wv;
  const __amdgpu_buffer_rsrc_t wr = mk_rsrc(myws + WS_OFF + (size_t)q * WS_PER, (u32)WS_PER);
  const __amdgpu_buffer_rsrc_t xr = mk_rsrc(myws + XS_OFF + (size_t)q * XS_PER, (u32)XS_PER);
  const __amdgpu_buffer_rsrc_t ors = mk_rsrc(myout, (u32)(MM * NN * 2));
  const __amdgpu_buffer_rsrc_t fr = mk_rsrc(myws + FLAG_OFF, (u32)(TF_OFF - FLAG_OFF));

  if (wv < 4) {
    const u32 mo = (u32)(((1024 * ri + 16 * r16 + (tid >> 4)) * NN + 512 * rj + 64 * g + 4 * (tid & 15)) * 2);
    __builtin_amdgcn_s_sleep(16);
    for (int i = 0; i < (gs & 3) * 4 + wv; ++i) __builtin_amdgcn_s_sleep(2);
#pragma unroll
    for (int qt = 0; qt < 4; ++qt) {
      __builtin_amdgcn_raw_buffer_store_b64(u32x2{SENT, SENT}, ors, mo + (u32)(qt * 256 * NN * 2), 0, AUX_SYS);
      __builtin_amdgcn_s_sleep(16);
    }
    wait_vm<0>();
    if (lane == 0) st_flag((u32*)(myws + AF_OFF + (size_t)r16 * FSTR) + g * 4 + wv, call);
  }
  u32x4 wf[8][4];
  bool ok = true;
  auto poll = [&](u32 fo) -> u32x4 { return __builtin_amdgcn_raw_buffer_load_b128(fr, fo, 0, AUX_SYS); };
  auto await = [&](u32 fo, u32x4 v, u32& i0, u32& i1) {
    u64 t0 = 0; u32 spins = 0;
    for (;;) {
      const u32 c0 = __builtin_amdgcn_readfirstlane(v.x), c1 = __builtin_amdgcn_readfirstlane(v.z);
      i0 = __builtin_amdgcn_readfirstlane(v.y); i1 = __builtin_amdgcn_readfirstlane(v.w);
      if (c0 == call && c1 == call && (i0 >> 22) == (call & 1023u) && (i1 >> 22) == (call & 1023u)) break;
      if ((++spins & 31u) == 0) {
        const u64 now = __builtin_amdgcn_s_memrealtime();
        if (t0 == 0) t0 = now;
        if (__builtin_amdgcn_readfirstlane(__hip_atomic_load(tmo, __ATOMIC_RELAXED, __HIP_MEMORY_SCOPE_AGENT)) == call || now - t0 > DEADLINE_TICKS) {
          __hip_atomic_store(tmo, call, __ATOMIC_RELAXED, __HIP_MEMORY_SCOPE_AGENT);
          ok = false; i0 = 0; i1 = 0;
          break;
        }
      }
      __builtin_amdgcn_s_sleep(PSLEEP);
      asm volatile("" ::: "memory");
      v = poll(fo);
    }
    asm volatile("" ::: "memory");
  };
  struct Planes { u32x4 l[4]; u32x2 n[4], e[4]; u32 i0, i1; };
  Planes P0;
  auto fetch = [&](Planes& P, const __amdgpu_buffer_rsrc_t& rs, u32 base, u32 stride) {
#pragma unroll
    for (int i = 0; i < 4; ++i) {
      P.l[i] = __builtin_amdgcn_raw_buffer_load_b128(rs, lane * 16, base + i * stride, AUX_LD);
      P.n[i] = __builtin_amdgcn_raw_buffer_load_b64(rs, 1024 + lane * 8, base + i * stride, AUX_LD);
    }
    if ((P.i0 | P.i1) & 0x3c0780u) {
#pragma unroll
      for (int i = 0; i < 4; ++i)
        P.e[i] = __builtin_amdgcn_raw_buffer_load_b64(rs, 1536 + lane * 8, base + i * stride, AUX_LD);
    }
  };
  auto decode = [&](Planes& P, auto&& out) {
#pragma unroll
    for (int i = 0; i < 4; ++i) {
      const u32 inf = ((i < 2 ? P.i0 : P.i1) >> (11 * (i & 1))) & 0x7ffu;
      u32x4 a, b;
      dec_unit(P.l[i], P.n[i], P.e[i], inf, lane, a, b);
      out(i, a, b);
    }
  };
  const u32 wfo = (u32)(WF_OFF - FLAG_OFF) + (u32)(g * 512 + q * 64);
  const u32 xfo0 = (u32)(XF_OFF - FLAG_OFF) + (u32)(r16 * 128 + q * 16);
#define TSG(i) do {} while (0)
#define TSQ(i) do {} while (0)
#define TSN(i) do {} while (0)
  Planes P1;
  auto start = [&](auto H, Planes& P) -> u32x4 {
    constexpr int h = decltype(H)::value;
    if (h < 4) fetch(P, wr, (u32)(((g * 8 + 4 * (h >> 1)) * 4 + 2 * (h & 1)) * 1024), 4096u);
    else fetch(P, xr, (u32)(r16 * 8192), 2048u);
    return poll(h < 3 ? wfo + (h + 1) * 16 : xfo0 + (h - 3) * 2048);
  };
  await(wfo, poll(wfo), P0.i0, P0.i1);
  u32x4 pv4 = start(IC<0>{}, P0);
  static_for<4>([&](auto H) {
    constexpr int h = decltype(H)::value;
    Planes& cur = (h & 1) ? P1 : P0;
    Planes& nxt = (h & 1) ? P0 : P1;
    const u32 c0 = __builtin_amdgcn_readfirstlane(pv4.x), c1 = __builtin_amdgcn_readfirstlane(pv4.z);
    nxt.i0 = __builtin_amdgcn_readfirstlane(pv4.y); nxt.i1 = __builtin_amdgcn_readfirstlane(pv4.w);
    const bool early = c0 == call && c1 == call && (nxt.i0 >> 22) == (call & 1023u) && (nxt.i1 >> 22) == (call & 1023u);
    asm volatile("" ::: "memory");
    const u32 fo = h < 3 ? wfo + (h + 1) * 16 : xfo0;
    if (early) pv4 = start(IC<h + 1>{}, nxt);
    else pv4 = poll(fo);
    __builtin_amdgcn_sched_barrier(0);
    decode(cur, [&](int i, u32x4 a, u32x4 b) { wf[4 * (h >> 1) + i][2 * (h & 1)] = a; wf[4 * (h >> 1) + i][2 * (h & 1) + 1] = b; });
    __builtin_amdgcn_sched_barrier(0);
    if (!early) {
      await(fo, pv4, nxt.i0, nxt.i1);
      pv4 = start(IC<h + 1>{}, nxt);
    }
    if (!(h & 1)) TSG(1 + (h >> 1));
  });
  bool have = true;
  asm volatile("" :: "v"(pv4));
#pragma unroll 1
  for (int qt = 0; qt < 4; ++qt) {
    const int rf = 16 * qt + r16;
    const u32 xfo = xfo0 + (u32)(qt * 2048);
    const u32 xfn = qt < 3 ? xfo + 2048 : xfo;
    if (!have) {
      const u32 c0 = __builtin_amdgcn_readfirstlane(pv4.x), c1 = __builtin_amdgcn_readfirstlane(pv4.z);
      P0.i0 = __builtin_amdgcn_readfirstlane(pv4.y); P0.i1 = __builtin_amdgcn_readfirstlane(pv4.w);
      if (!(c0 == call && c1 == call && (P0.i0 >> 22) == (call & 1023u) && (P0.i1 >> 22) == (call & 1023u)))
        await(xfo, poll(xfo), P0.i0, P0.i1);
      asm volatile("" ::: "memory");
      fetch(P0, xr, (u32)(rf * 8192), 2048u);
      pv4 = poll(xfn);
      asm volatile("" :: "v"(pv4));
    }
    TSN(3 + 3 * qt);
    u32x4 xf[8];
    __builtin_amdgcn_sched_barrier(0);
    decode(P0, [&](int i, u32x4 a, u32x4 b) { xf[2 * i] = a; xf[2 * i + 1] = b; });
    __builtin_amdgcn_sched_barrier(0);
    auto look = [&]() {
      const u32 c0 = __builtin_amdgcn_readfirstlane(pv4.x), c1 = __builtin_amdgcn_readfirstlane(pv4.z);
      const u32 j0 = __builtin_amdgcn_readfirstlane(pv4.y), j1 = __builtin_amdgcn_readfirstlane(pv4.w);
      have = qt < 3 && c0 == call && c1 == call && (j0 >> 22) == (call & 1023u) && (j1 >> 22) == (call & 1023u);
      asm volatile("" ::: "memory");
      if (have) {
        P0.i0 = j0; P0.i1 = j1;
        fetch(P0, xr, (u32)((rf + 16) * 8192), 2048u);
      }
      pv4 = poll(have && qt < 2 ? xfn + 2048 : xfn);
    };
    f32x4 acc[4];
#pragma unroll
    for (int f = 0; f < 4; ++f) acc[f] = f32x4{0.f, 0.f, 0.f, 0.f};
#pragma unroll
    for (int s = 0; s < 8; ++s)
#pragma unroll
      for (int f = 0; f < 4; ++f)
        acc[f] = __builtin_amdgcn_mfma_f32_16x16x32_bf16(
            __builtin_bit_cast(bf16x8, wf[s][f]), __builtin_bit_cast(bf16x8, xf[s]), acc[f], 0, 0, 0);
    __builtin_amdgcn_sched_barrier(0);
    look();
    __builtin_amdgcn_sched_barrier(0);
    char* const pb = smem + (qt & 1) * PBUF;
    u32* const sbad = (u32*)(pb + 8 * PART_BYTES);
    {
      char* part = pb + wv * PART_BYTES;
      const int m = lane & 15, q4 = lane >> 4;
#pragma unroll
      for (int reg = 0; reg < 4; ++reg) {
        const f32x4 v = {acc[0][reg], acc[1][reg], acc[2][reg], acc[3][reg]};
        *(f32x4*)(part + m * PROW + q4 * 64 + reg * 16) = v;
      }
      if (lane == 0) sbad[wv] = ok ? 0u : 1u;
    }
    asm volatile("s_waitcnt lgkmcnt(0)" ::: "memory");
    __builtin_amdgcn_s_barrier();
    asm volatile("" ::: "memory");
    TSN(4 + 3 * qt);
    {
      const int orow = tid >> 5, oseg = tid & 31;
      f32x2 pv[8];
#pragma unroll
      for (int k = 0; k < 8; ++k) pv[k] = *(const f32x2*)(pb + k * PART_BYTES + orow * PROW + oseg * 8);
      const f32x2 s0 = ((pv[0] + pv[1]) + (pv[2] + pv[3])) + ((pv[4] + pv[5]) + (pv[6] + pv[7]));
      const u32x4 bad0 = *(const u32x4*)sbad, bad1 = *(const u32x4*)(sbad + 4);
      const bool bad = (bad0.x | bad0.y | bad0.z | bad0.w | bad1.x | bad1.y | bad1.z | bad1.w) != 0u;
      u32 o = pack_bf16(silu(s0[0]), silu(s0[1]));
      if (bad || o == SENT) o = 0x7fc07fc0u;
      const u32 ooff = (u32)(((1024 * ri + 16 * rf + orow) * NN + 512 * rj + 64 * g + 2 * oseg) * 2);
      __builtin_amdgcn_raw_buffer_store_b32(o, ors, ooff, 0, AUX_SYS);
    }
    TSN(5 + 3 * qt);
  }
  if (tid == 0) __hip_atomic_store(cntp, call, __ATOMIC_RELAXED, __HIP_MEMORY_SCOPE_AGENT);
  TS_FLUSH();
}

extern "C" void kernel_launch(int rank, int nranks, ncclComm_t comm,
                              hipStream_t stream,
                              void* const* d_in, const int* in_sizes, int n_in,
                              void* d_out, int out_size,
                              const int* in_dtypes, int out_dtype,
                              void* const* d_out_all,
                              void* const* d_ws_all, size_t ws_bytes) {
  (void)comm; (void)in_sizes; (void)n_in; (void)d_out; (void)out_size; (void)in_dtypes; (void)out_dtype;
  if (nranks != NR || (unsigned)rank >= (unsigned)NR || ws_bytes < WS_NEEDED) {
    fprintf(stderr, "unsupported configuration\n");
    return;
  }
  Params p;
  p.x = (const u16*)d_in[0];
  p.w = (const u16*)d_in[1];
  constexpr unsigned QUAD1 = 0xb2u;
  const unsigned myq = (QUAD1 >> rank) & 1u;
  int lo = 0, hi = 4;
  p.dmap = 0;
  for (int d = 0; d < NR; ++d) {
    const int x = (((QUAD1 >> d) & 1u) == myq) ? lo++ : hi++;
    p.dmap |= (u32)d << (4 * x);
    p.out[x] = (char*)d_out_all[d];
    p.ws[x] = (char*)d_ws_all[d];
  }
  p.rank = rank;
  p.myws = (char*)d_ws_all[rank];
  p.myout = (char*)d_out_all[rank];
  p.pad_[0] = p.pad_[1] = 0;
  gemm_ar_silu<<<GRID, 512, 0, stream>>>(p);
  HIP_CHECK(hipGetLastError());
}
```
